```python
import numpy as np

kernel_name = "gpu_gemm_a2a_m4096_k4096_n4096_e5m2e5m2_dequant"


ABSMAX_RTOL = 0.05

COMPUTE_DTYPE = "float32"
EPILOGUE = "dequant"
IN_QUANT = None
OUT_QUANT = None
IN_FP8 = "e5m2e5m2"
IN_DTYPES = {"x": "e5m2", "w_mat": "e5m2",
             "scale_x": "float32", "scale_w": "float32"}
WORLD_SIZE = 8

def _snap_fp8(x, mant_bits, max_abs, min_normal):
    a = np.abs(x.astype(np.float64))
    _m, _e = np.frexp(a)
    q = np.where(a >= min_normal, np.ldexp(1.0, _e - 1 - mant_bits),
                 min_normal * 2.0 ** -mant_bits)
    snapped = np.minimum(np.round(a / q) * q, max_abs)
    return (np.sign(x) * snapped).astype(np.float32)

def setup_inputs():

    x = _snap_fp8(np.random.randn(4096, 4096).astype(np.float32), 2, 57344.0, 2.0 ** -14)
    w_mat = _snap_fp8(np.random.randn(4096, 4096).astype(np.float32), 2, 57344.0, 2.0 ** -14)
    scale_x = (0.02 * (1.0 + np.random.rand(1))).astype(np.float32)
    scale_w = (0.02 * (1.0 + np.random.rand(1))).astype(np.float32)
    return {"x": x, "w_mat": w_mat, "scale_x": scale_x, "scale_w": scale_w}

def reference(x, w_mat, scale_x, scale_w):
    acc = (x @ w_mat).astype(np.float32)
    y = (acc * (scale_x * scale_w)).astype(np.float32)
    return y
```

```cpp
#include <hip/hip_runtime.h>
#include <rccl/rccl.h>
#include <cstdio>
#include <cstdlib>

typedef unsigned u32x4 __attribute__((ext_vector_type(4)));
typedef int i32x2 __attribute__((ext_vector_type(2)));
typedef int i32x4 __attribute__((ext_vector_type(4)));
typedef int i32x8 __attribute__((ext_vector_type(8)));
typedef float f32x4 __attribute__((ext_vector_type(4)));
#define LDSP __attribute__((address_space(3)))

constexpr int NR = 8, MS = 512, KD = 4096, ND = 4096, NS = 512;
constexpr int RING = 32768;
constexpr int PART_OFF = 4 * RING;
constexpr int LDS_BYTES = PART_OFF + 2 * 16384;
constexpr unsigned long long DEADLINE = 20000000ull;
constexpr size_t WS_CALL = 0, WS_TMO = 128, WS_CNT = 4096;
#define STAMP(i)

struct Args {
  float* out[NR];
  unsigned char* ws[NR];
  const unsigned char* x;
  const unsigned char* w;
  const float* sx;
  const float* sw;
  int rank;
  int delay[4];
  int eager[4];
  int rot, pf, win, pad;
};

__device__ __forceinline__ void glds16(i32x4 srd, unsigned voff, unsigned soff, unsigned lds) {
  asm volatile("s_mov_b32 m0, %3\n\ts_nop 0\n\tbuffer_load_dwordx4 %0, %1, %2 offen lds" ::"v"(voff), "s"(srd), "s"(soff), "s"(lds) : "memory");
}
#define WAITV(n) asm volatile("s_waitcnt vmcnt(%0)" ::"n"(n) : "memory")
#define WAITL0() asm volatile("s_waitcnt lgkmcnt(0)" ::: "memory")
#define SB() __builtin_amdgcn_sched_barrier(0)

__device__ __forceinline__ i32x4 make_srd(const void* p, unsigned bytes) {
  unsigned long long a = (unsigned long long)p;
  i32x4 r; r.x = (int)(unsigned)a; r.y = (int)((unsigned)(a >> 32) & 0xffffu); r.z = (int)bytes; r.w = 0x00020000; return r;
}

__global__ __launch_bounds__(256, 1) void gemm_a2a(Args a) {
  extern __shared__ __attribute__((aligned(16))) unsigned char smem[];
  const int b = blockIdx.x, xcd = b & 7, j = b >> 3, set = j >> 3, ph = j & 1, qr = (j >> 1) & 3;
  const int dest = (a.rank + 1 + xcd + set * a.rot) & 7, blk = set;
  const int c0 = blk * 128 + ph * 64, n0 = dest * NS + c0, r0 = qr * 128;
  const int w = __builtin_amdgcn_readfirstlane(threadIdx.x >> 6), l = threadIdx.x & 63;
  STAMP(0);
  const i32x4 wsrd = make_srd(a.w, KD * ND);
  if (set > 0) {
    int dl = a.delay[set];
    unsigned long long t0 = __builtin_amdgcn_s_memrealtime();
    if (a.pf) {
      const i32x4 xall = make_srd(a.x, MS * KD);
      const int pw = (j - 8) * 4 + w;
      unsigned dummy[6];
#pragma unroll
      for (int r = 0; r < 6; ++r) {
        int isw, idx, c;
        if (r == 0) { isw = pw < 64; idx = 0; c = isw ? pw : pw - 64; }
        else if (r == 1) { isw = pw >= 32; idx = 1; c = isw ? pw - 32 : pw; }
        else if (r == 2) { isw = 0; idx = 2 + (pw >> 5); c = pw & 31; }
        else if (r == 3) { isw = pw >= 32; idx = isw ? 2 : 5; c = isw ? pw - 32 : pw; }
        else if (r == 4) { isw = pw >= 64; idx = isw ? 3 : 6 + (pw >> 5); c = pw & 31; }
        else { isw = 1; idx = 3; c = 32 + pw; }
        if (r == 5 && pw >= 32) break;
        if (r >= a.pf) break;
        unsigned off;
        if (isw) { int pd = (a.rank + 1 + xcd + idx * a.rot) & 7; off = (unsigned)(c * 64 + l) * ND + pd * NS + idx * 128; }
        else { int row = (c >> 3) * 128 + idx * 16 + (c & 7) * 2 + (l >> 5); off = (unsigned)row * KD + (l & 31) * 128; }
        if (isw) asm volatile("buffer_load_dword %0, %1, %2, 0 offen" : "=v"(dummy[r]) : "v"(off), "s"(wsrd) : "memory");
        else asm volatile("buffer_load_dword %0, %1, %2, 0 offen" : "=v"(dummy[r]) : "v"(off), "s"(xall) : "memory");
        WAITV(0);
      }
    }
    while (__builtin_amdgcn_s_memrealtime() - t0 < (unsigned long long)dl) __builtin_amdgcn_s_sleep(1);
  }
  STAMP(1);
  const i32x4 xsrd = make_srd(a.x + (size_t)r0 * KD, 128 * KD);
  const unsigned xoffE = (l >> 3) * KD + (((l & 7) ^ ((l >> 4) & 3)) << 4) + w * 1024;
  const unsigned xoffO = (xoffE ^ 64) + 8 * KD;
  const unsigned woffE = (l >> 2) * ND + (((l & 3) ^ ((l >> 4) & 1)) << 4) + n0;
  const unsigned woffO = woffE ^ 32;
  const unsigned wk0 = (unsigned)w * 1024 * ND;
  const unsigned ring = (unsigned)(size_t)(LDSP unsigned char*)smem + w * RING;

  auto issue_x = [&](int g, int h, int slot) {
#pragma unroll
    for (int i = 0; i < 8; ++i)
      glds16(xsrd, (i & 1) ? xoffO : xoffE, g * 16 * KD + h * 512 + (i >> 1) * 128, ring + slot * 8192 + i * 1024);
  };
  auto issue_w = [&](int kb, int slot) {
#pragma unroll
    for (int i = 0; i < 8; ++i)
      glds16(wsrd, (i & 1) ? woffO : woffE, wk0 + (unsigned)(kb * 128 + i * 16) * ND, ring + slot * 8192 + i * 1024);
  };
  issue_x(0, 0, 0); issue_x(0, 1, 1); issue_w(0, 2); issue_w(1, 3);

  i32x8 xf[8];
  i32x8 wf[8][4];
  LDSP unsigned char* rd[4];
#pragma unroll
  for (int ct = 0; ct < 4; ++ct)
    rd[ct] = (LDSP unsigned char*)smem + w * RING + (16 * (l >> 4) + ((l & 15) >> 1)) * 64 + ((ct ^ ((l >> 3) & 3)) << 4) + (l & 1) * 8;
  const int xr = (l & 15) * 128 + (((l >> 4) ^ ((l >> 1) & 7)) << 4);
  LDSP unsigned char* xrd0 = (LDSP unsigned char*)smem + w * RING + xr;
  LDSP unsigned char* xrd1 = (LDSP unsigned char*)smem + w * RING + (xr ^ 64);
  auto read_w = [&](int kb, int slot) {
#pragma unroll
    for (int ct = 0; ct < 4; ++ct) {
      i32x2 t[4];
#pragma unroll
      for (int u = 0; u < 4; ++u)
        t[u] = __builtin_amdgcn_ds_read_tr8_b64_v2i32((LDSP i32x2*)(rd[ct] + slot * 8192 + (u & 1) * 512 + (u >> 1) * 4096));
      wf[kb][ct] = i32x8{t[0].x, t[0].y, t[1].x, t[1].y, t[2].x, t[2].y, t[3].x, t[3].y};
    }
  };
  auto read_x = [&](int h, int slot) {
#pragma unroll
    for (int k = 0; k < 4; ++k) {
      i32x4 lo = *(LDSP i32x4*)(xrd0 + slot * 8192 + k * 2048), hi = *(LDSP i32x4*)(xrd1 + slot * 8192 + k * 2048);
      xf[h * 4 + k] = i32x8{lo.x, lo.y, lo.z, lo.w, hi.x, hi.y, hi.z, hi.w};
    }
  };
  f32x4 acc[4];
  auto mma = [&](int kb) {
#pragma unroll
    for (int ct = 0; ct < 4; ++ct)
      acc[ct] = __builtin_amdgcn_mfma_scale_f32_16x16x128_f8f6f4(wf[kb][ct], xf[kb], kb == 0 ? f32x4{0.f, 0.f, 0.f, 0.f} : acc[ct], 1, 1, 0, 0, 0, 0);
  };
  typedef const __attribute__((address_space(4))) float* cfp;
  const float scale = *(cfp)a.sx * *(cfp)a.sw;
  const unsigned ooff = ((l & 15) * NS + c0 + w * 16 + (l >> 4) * 4) * 4;
  f32x4 res[8];
  const int eager = a.eager[set];
  const auto ors = __builtin_amdgcn_make_buffer_rsrc((void*)(a.out[dest] + (size_t)(a.rank * MS + r0) * NS), 0, 128 * NS * 4, 0x00020000);
  const auto ors_e = __builtin_amdgcn_make_buffer_rsrc((void*)(a.out[dest] + (size_t)(a.rank * MS + r0) * NS), 0, eager ? 128 * NS * 4 : 0, 0x00020000);
  auto reduce_store = [&](int g) {
    LDSP unsigned char* pb = (LDSP unsigned char*)smem + PART_OFF + (g & 1) * 16384 + l * 16;
#pragma unroll
    for (int ct = 0; ct < 4; ++ct) *(LDSP f32x4*)(pb + ct * 4096 + w * 1024) = acc[ct];
    WAITL0();
    __builtin_amdgcn_s_barrier();
    asm volatile("" ::: "memory");
    f32x4 p0 = *(LDSP f32x4*)(pb + w * 4096), p1 = *(LDSP f32x4*)(pb + w * 4096 + 1024);
    f32x4 p2 = *(LDSP f32x4*)(pb + w * 4096 + 2048), p3 = *(LDSP f32x4*)(pb + w * 4096 + 3072);
    f32x4 s = ((p0 + p1) + (p2 + p3)) * scale;
    res[g] = s;
    __builtin_amdgcn_raw_buffer_store_b128(__builtin_bit_cast(u32x4, s), ors_e, ooff, g * 16 * NS * 4, 17);
  };

  WAITV(24); read_x(0, 0); WAITL0(); issue_w(2, 0); SB();
  WAITV(24); read_x(1, 1); WAITL0(); issue_w(3, 1); SB();
  WAITV(24); read_w(0, 2); WAITL0(); issue_w(4, 2); mma(0); SB();
  WAITV(24); read_w(1, 3); WAITL0(); issue_w(5, 3); mma(1); SB();
  WAITV(24); read_w(2, 0); WAITL0(); issue_w(6, 0); mma(2); SB();
  WAITV(24); read_w(3, 1); WAITL0(); issue_w(7, 1); mma(3); SB();
  WAITV(24); read_w(4, 2); WAITL0(); issue_x(1, 0, 2); mma(4); SB();
  WAITV(24); read_w(5, 3); WAITL0(); issue_x(1, 1, 3); mma(5); SB();
  WAITV(24); read_w(6, 0); mma(6); SB();
  WAITV(16); read_w(7, 1); mma(7); SB();
  STAMP(2);
  reduce_store(0);
  STAMP(3);
  SB(); issue_x(2, 0, 0); issue_x(2, 1, 1); SB();
#pragma unroll
  for (int g = 1; g < 8; ++g) {
    const int s0 = (g & 1) ? 2 : 0;
    const int more = (g + 1 < 8) ? 16 : 0;
    const int na = g == 1 ? 25 : g == 2 ? 8 + more + 1 : 8 + 1 + more + 1;
    WAITV(na); read_x(0, s0); WAITL0();
    if (g + 2 < 8) issue_x(g + 2, 0, s0);
    mma(0); mma(1); mma(2); mma(3); SB();
    const int nb = na - 8 + ((g + 2 < 8) ? 8 : 0);
    WAITV(nb); read_x(1, s0 + 1); WAITL0();
    if (g + 2 < 8) issue_x(g + 2, 1, s0 + 1);
    mma(4); mma(5); mma(6); mma(7); SB();
    reduce_store(g); SB();
  }
  STAMP(4);
  if (!eager) {
#pragma unroll
    for (int g = 0; g < 8; ++g) {
      __builtin_amdgcn_raw_buffer_store_b128(__builtin_bit_cast(u32x4, res[g]), ors, ooff, g * 16 * NS * 4, 17);
      if (a.win == 1) WAITV(0); else if (a.win == 2) WAITV(1);
    }
  }
  WAITV(0);
  __syncthreads();
  STAMP(5);
  unsigned char* myws = a.ws[a.rank];
  if (threadIdx.x == 0 && dest != a.rank)
    __hip_atomic_fetch_add((unsigned*)(a.ws[dest] + WS_CNT + a.rank * 128), 1u, __ATOMIC_RELAXED, __HIP_MEMORY_SCOPE_SYSTEM);
  if (set == 3 && dest == a.rank && (j & 7) == 7 && w == 0) {
    unsigned call = 0;
    if (l == 0) call = __hip_atomic_fetch_add((unsigned*)(myws + WS_CALL), 1u, __ATOMIC_RELAXED, __HIP_MEMORY_SCOPE_AGENT) + 1;
    call = __builtin_amdgcn_readfirstlane(call);
    if (l < NR && l != a.rank) {
      unsigned* f = (unsigned*)(myws + WS_CNT + l * 128);
      unsigned long long t0 = __builtin_amdgcn_s_memrealtime();
      while (__hip_atomic_load(f, __ATOMIC_RELAXED, __HIP_MEMORY_SCOPE_SYSTEM) < call * 32u) {
        if (__builtin_amdgcn_s_memrealtime() - t0 > DEADLINE) {
          __hip_atomic_store((unsigned*)(myws + WS_TMO), call, __ATOMIC_RELAXED, __HIP_MEMORY_SCOPE_AGENT);
          a.out[a.rank][(size_t)(a.rank * MS + r0) * NS + c0] = __builtin_nanf("");
          break;
        }
        __builtin_amdgcn_s_sleep(1);
      }
    }
  }
}

extern "C" void kernel_launch(int rank, int nranks, ncclComm_t comm, hipStream_t stream, void* const* d_in, const int* in_sizes, int n_in,
                              void* d_out, int out_size, const int* in_dtypes, int out_dtype, void* const* d_out_all, void* const* d_ws_all, size_t ws_bytes) {
  static bool init = false;
  if (!init) { (void)hipFuncSetAttribute((const void*)gemm_a2a, hipFuncAttributeMaxDynamicSharedMemorySize, LDS_BYTES); init = true; }
  Args a;
  for (int p = 0; p < NR; ++p) { a.out[p] = (float*)d_out_all[p]; a.ws[p] = (unsigned char*)d_ws_all[p]; }
  a.x = (const unsigned char*)d_in[0]; a.w = (const unsigned char*)d_in[1]; a.sx = (const float*)d_in[2]; a.sw = (const float*)d_in[3];
  a.rank = rank;
  auto env = [](const char* n, int d) { return d; };
  a.delay[0] = 0; a.delay[1] = env("D1", 0); a.delay[2] = env("D2", 0); a.delay[3] = env("D3", 0);
  int eg = env("EAGER", 15); for (int i = 0; i < 4; ++i) a.eager[i] = (eg >> i) & 1;
  a.rot = env("ROT", 0); a.pf = env("PF", 0); a.win = env("WIN", 0); a.pad = 0;
  gemm_a2a<<<256, 256, LDS_BYTES, stream>>>(a);
}
```

```cpp
#include <hip/hip_runtime.h>
#include <rccl/rccl.h>
#include <cstdio>
#include <cstdlib>

typedef unsigned u32x4 __attribute__((ext_vector_type(4)));
typedef int i32x2 __attribute__((ext_vector_type(2)));
typedef int i32x4 __attribute__((ext_vector_type(4)));
typedef int i32x8 __attribute__((ext_vector_type(8)));
typedef int i32x16 __attribute__((ext_vector_type(16)));
typedef float f32x4 __attribute__((ext_vector_type(4)));
typedef float f32x32 __attribute__((ext_vector_type(32)));
typedef unsigned long long u64;
#define LDSP __attribute__((address_space(3)))
#define CONSTP __attribute__((address_space(4)))
typedef __attribute__((address_space(1))) unsigned gu32;
typedef __attribute__((address_space(1))) float gf32;
typedef __attribute__((address_space(1))) u64 gu64;

constexpr int NR = 8, MS = 512, KD = 4096, ND = 4096, NS = 512;
constexpr int RING = 32768;
constexpr int PART_OFF = 4 * RING;
constexpr int LDS_BYTES = PART_OFF + 2 * 16384;
constexpr u64 DEADLINE = 20000000ull;
constexpr unsigned WS_DONE = 0, WS_TMO = 128, WS_OCNT = 4096, WS_XCNT = 8192, WS_XS = 65536;
constexpr unsigned XS_SRC = 64 * KD, XS_PAR = NR * XS_SRC;
constexpr unsigned OOB = 0x40000000u;
#define STAMP(i)
#define LSTAMP(i)
#define STEP(i)

struct Args {
  const void* x; const void* w; void* myws; const void* sx; const void* sw;
  int rank, cfg, d12, d3l, fmask, win;
  struct { void* out; void* ws; } peer[NR];
};

#define WAITV(n) asm volatile("s_waitcnt vmcnt(%0)" ::"n"(n) : "memory")
#define WAITL(n) asm volatile("s_waitcnt lgkmcnt(%0)" ::"n"(n) : "memory")
#define SB() __builtin_amdgcn_sched_barrier(0)
#define BAR() do { __builtin_amdgcn_s_barrier(); asm volatile("" ::: "memory"); } while (0)

template <int OFF>
__device__ __forceinline__ void dma_w(i32x4 srd, unsigned voff, unsigned soff, unsigned lds) {
  asm volatile("s_mov_b32 m0, %4\n\ts_nop 0\n\tbuffer_load_dwordx4 %0, %1, %2 offen offset:%3 lds" ::"v"(voff), "s"(srd), "s"(soff), "n"(OFF), "s"(lds) : "memory");
}
template <int OFF>
__device__ __forceinline__ void dma_x(i32x4 srd, unsigned voff, unsigned soff, unsigned lds) {
  asm volatile("s_mov_b32 m0, %4\n\ts_nop 0\n\tbuffer_load_dwordx4 %0, %1, %2 offen offset:%3 sc0 sc1 lds" ::"v"(voff), "s"(srd), "s"(soff), "n"(OFF), "s"(lds) : "memory");
}
__device__ __forceinline__ void dma_touch(i32x4 srd, unsigned voff, unsigned soff, unsigned lds) {
  asm volatile("s_mov_b32 m0, %3\n\ts_nop 0\n\tbuffer_load_dword %0, %1, %2 offen lds" ::"v"(voff), "s"(srd), "s"(soff), "s"(lds) : "memory");
}
__device__ __forceinline__ i32x4 make_srd(u64 a, unsigned bytes) {
  i32x4 r; r.x = (int)(unsigned)a; r.y = (int)((unsigned)(a >> 32) & 0xffffu); r.z = (int)bytes; r.w = 0x00020000; return r;
}
__device__ __forceinline__ void store16(i32x4 srd, unsigned voff, unsigned soff, u32x4 v) {
  asm volatile("buffer_store_dwordx4 %0, %1, %2, %3 offen sc0 sc1\n\ts_nop 1" ::"v"(v), "v"(voff), "s"(srd), "s"(soff) : "memory");
}
__device__ __forceinline__ void store16_plain(i32x4 srd, unsigned voff, unsigned soff, u32x4 v) {
  asm volatile("buffer_store_dwordx4 %0, %1, %2, %3 offen\n\ts_nop 1" ::"v"(v), "v"(voff), "s"(srd), "s"(soff) : "memory");
}
__device__ __forceinline__ bool wait_counters(u64 myws, unsigned off, unsigned target, unsigned call, int rank, int l) {
  bool bad = false;
  if (l < NR && l != rank) {
    gu32* f = (gu32*)(myws + off + l * 128);
    gu32* tmo = (gu32*)(myws + WS_TMO);
    const u64 t0 = __builtin_amdgcn_s_memrealtime();
    while (__hip_atomic_load(f, __ATOMIC_RELAXED, __HIP_MEMORY_SCOPE_SYSTEM) < target) {
      if (__hip_atomic_load(tmo, __ATOMIC_RELAXED, __HIP_MEMORY_SCOPE_AGENT) == call || __builtin_amdgcn_s_memrealtime() - t0 > DEADLINE) {
        __hip_atomic_store(tmo, call, __ATOMIC_RELAXED, __HIP_MEMORY_SCOPE_AGENT);
        bad = true;
        break;
      }
      __builtin_amdgcn_s_sleep(1);
    }
  }
  return __builtin_amdgcn_ballot_w64(bad) != 0;
}

__global__ __launch_bounds__(256, 1) void gemm_a2a(Args args) {
  extern __shared__ __attribute__((aligned(16))) unsigned char smem[];
  STAMP(0);
  const u64 tstart = __builtin_amdgcn_s_memrealtime();
  const int b = blockIdx.x, xcd = b & 7, j = b >> 3, blk = j >> 3, ph = j & 1, qr = (j >> 1) & 3;
  const CONSTP unsigned char* kp = (const CONSTP unsigned char*)__builtin_amdgcn_kernarg_segment_ptr();
  i32x16 hv = *(const CONSTP i32x16*)kp;
  i32x4 pv = *(const CONSTP i32x4*)(kp + 64 + xcd * 16);
  asm volatile("" : "+s"(hv), "+s"(pv));
  auto mk64 = [](int lo, int hi) { return (unsigned)lo | ((u64)(unsigned)hi << 32); };
  const u64 ax = mk64(hv[0], hv[1]), aw = mk64(hv[2], hv[3]), myws = mk64(hv[4], hv[5]), asx = mk64(hv[6], hv[7]), asw = mk64(hv[8], hv[9]);
  const int rank = hv[10], nR = hv[11] & 7, pf = (hv[11] >> 4) & 1, pwin = hv[11] >> 8, d12 = hv[12], d3l = hv[13], fmask = hv[14], win = hv[15] & 15, dl0 = hv[15] >> 4;
  const u64 dout = mk64(pv.x, pv.y), dws = mk64(pv.z, pv.w);
  const int dest = (rank + 1 + xcd) & 7;
  const bool local = xcd == 7;
  const int c0 = blk * 128 + ph * 64, n0 = dest * NS + c0, r0 = qr * 128;
  const int w = __builtin_amdgcn_readfirstlane(threadIdx.x >> 6), l = threadIdx.x & 63;
  const unsigned lds0 = (unsigned)(size_t)(LDSP unsigned char*)smem;
  const unsigned ring = lds0 + w * RING;
  const i32x4 wsrd = make_srd(aw, KD * ND);
  unsigned call = 0;

  if (blk > 0 || local) {
    const i32x4 xall = make_srd(ax, MS * KD);
    auto until = [&](int ticks) { while ((int)(unsigned)(__builtin_amdgcn_s_memrealtime() - tstart) < ticks) __builtin_amdgcn_s_sleep(1); };
    const unsigned pfs = pf ? 0u : OOB, scr = lds0 + PART_OFF + w * 4096;
    auto pf_w = [&](int pnl, int c) {
      const int k = ((c >> 1) & 3) * 1024 + (c >> 3) * 128 + (c & 1) * 64 + l;
      dma_touch(wsrd, (unsigned)k * ND + dest * NS + pnl * 128, pfs, scr);
    };
    auto pf_x = [&](int g, int c) {
      const int row = (c >> 3) * 128 + g * 16 + (c & 7) * 2 + (l >> 5);
      dma_touch(xall, (unsigned)row * KD + (l & 31) * 128, pfs, scr);
    };
    const int pw = (j & 7) * 4 + w;
    const int tgo = local ? dl0 + (d3l >> 16) * blk : blk == 1 ? (d12 & 0xffff) : blk == 2 ? (d12 >> 16) : (d3l & 0xffff);
    if (local) {
      unsigned done;
      asm volatile("global_load_dword %0, %1, %2 sc1\n\ts_waitcnt vmcnt(0)" : "=&v"(done) : "v"(WS_DONE), "s"(myws) : "memory");
      call = __builtin_amdgcn_readfirstlane(done) / 256u + 1u;
      if (blk >= 2) { until(dl0 - 100); pf_x(blk - 2, pw); }
    } else if (blk == 3) {
      unsigned done;
      u32x4 v[8];
      const unsigned poff = (pw >> 1) * KD + (pw & 1) * 2048 + l * 16;
      auto ld2 = [&](int i) {
        asm volatile("buffer_load_dwordx4 %0, %2, %3, %4 offen\n\tbuffer_load_dwordx4 %1, %2, %3, %4 offen offset:1024"
                     : "=&v"(v[2 * i]), "=&v"(v[2 * i + 1]) : "v"(poff), "s"(xall), "s"(i < nR ? (unsigned)((3 - i) * 128 + 112) * KD : OOB) : "memory");
      };
      asm volatile("global_load_dword %0, %1, %2 sc1" : "=&v"(done) : "v"(WS_DONE), "s"(myws) : "memory");
      ld2(0);
      pf_x(1, pw);
      ld2(1); ld2(2); ld2(3);
      asm volatile("s_waitcnt vmcnt(7)" : "+v"(done), "+v"(v[0]), "+v"(v[1])::"memory");
      call = __builtin_amdgcn_readfirstlane(done) / 256u + 1u;
      const i32x4 psrd = make_srd(dws + WS_XS + (call & 1) * XS_PAR + rank * XS_SRC, nR * 16 * KD);
#pragma unroll
      for (int i = 0; i < 4; ++i) {
        if (i) asm volatile("s_waitcnt vmcnt(6)" : "+v"(v[2 * i]), "+v"(v[2 * i + 1])::"memory");
        if (i && pwin == 2) WAITV(1);
        if (i && pwin == 3) WAITV(2);
        store16(psrd, poff, i * 16 * KD, v[2 * i]);
        if (i && pwin == 2) WAITV(1);
        if (i && pwin == 3) WAITV(2);
        store16(psrd, poff + 1024, i * 16 * KD, v[2 * i + 1]);
        if (i == 0) STAMP(8);
      }
      WAITV(0);
      BAR();
      STAMP(9);
      if (threadIdx.x == 0 && nR)
        __hip_atomic_fetch_add((gu32*)(dws + WS_XCNT + rank * 128), 1u, __ATOMIC_RELAXED, __HIP_MEMORY_SCOPE_SYSTEM);
    } else if (blk == 2) {
      pf_x(0, pw);
#pragma unroll
      for (int q = 0; q < 4; ++q) pf_w(pw >> 4, q * 16 + (pw & 15));
      pf_x(2, pw);
    }
    if (blk >= 2 || local) { until(tgo - 100); pf_w(blk, pw); pf_w(blk, 32 + pw); }
    until(tgo);
  }
  STAMP(1);

  unsigned xo[4], wo[4];
#pragma unroll
  for (int i = 0; i < 4; ++i) {
    xo[i] = 4096 + w * 1024 + ((l >> 3) + (i & 1) * 8) * KD + (((l & 7) ^ ((l >> 4) & 3) ^ ((i & 1) << 2)) << 4) + (i >> 1) * 128 - i * 1024;
    wo[i] = (w * 1024 + (l >> 2) + i * 16) * ND + (((l & 3) ^ ((l >> 4) & 1) ^ ((i & 1) << 1)) << 4) + n0 - i * 1024;
  }
  auto dmax = [&](i32x4 srd, int i, unsigned so, unsigned slot) {
    const unsigned s = so + (i >> 2) * 256, m = slot + (i >> 2) * 4096;
    switch (i & 3) { case 0: dma_x<0>(srd, xo[0], s, m); break; case 1: dma_x<1024>(srd, xo[1], s, m); break; case 2: dma_x<2048>(srd, xo[2], s, m); break; default: dma_x<3072>(srd, xo[3], s, m); }
  };
  const int nown = (!local && qr >= 4 - nR) ? 7 : 8;
  const int ng = local ? 8 + (7 * nR + 3 - qr) / 4 : nown;
  const u64 xown = ax + (unsigned)r0 * KD - 4096;
  const u64 xfor = myws + (WS_XS - 4096) + (call & 1) * XS_PAR;
  const unsigned fm = local ? 0xffffu : (unsigned)(fmask >> (blk * 8)) & 0xffu;
  unsigned t_lo, t_hi, t_rec, t_row, t_now;
  auto make_tables = [&]() {
    const int g = l, f = qr + 4 * (g - 8), i = (f * 37) >> 8, src = (rank + 1 + f - 7 * i) & 7;
    const u64 a = g < 8 ? xown + (unsigned)g * 16 * KD : xfor + (unsigned)(src * XS_SRC + i * 16 * KD);
    t_lo = (unsigned)a; t_hi = (unsigned)(a >> 32) & 0xffffu; t_rec = g < ng ? 17 * KD : 0;
    t_row = (unsigned)(g < 8 ? rank * MS + r0 + g * 16 : src * MS + (3 - i) * 128 + 112) * (NS * 4);
    t_now = ((fm >> (g & 15)) & 1) ? t_row : OOB;
    asm volatile("" : "+v"(t_lo), "+v"(t_hi), "+v"(t_rec), "+v"(t_row), "+v"(t_now));
  };
  auto group_srd = [&](int g) {
    i32x4 r = {__builtin_amdgcn_readlane((int)t_lo, g), __builtin_amdgcn_readlane((int)t_hi, g), __builtin_amdgcn_readlane((int)t_rec, g), 0x00020000};
    asm volatile("s_nop 4" : "+s"(r));
    return r;
  };
  auto lane_s = [&](unsigned t, int g) { unsigned r = __builtin_amdgcn_readlane((int)t, g); asm volatile("s_nop 4" : "+s"(r)); return r; };
  const i32x4 x03 = make_srd(xown, 65 * KD);
  auto fill1 = [&](int f, int i) {
    const unsigned m = ring + (f & 3) * 8192;
    if (f < 4 || f >= 12) {
      const int h = f < 4 ? f : f - 8;
      dmax(x03, i, (h >> 1) * 16 * KD + (h & 1) * 512, m);
    } else {
      const unsigned so = (unsigned)((f - 4) * 128 + (i >> 2) * 64) * ND, mm_ = m + (i >> 2) * 4096;
      switch (i & 3) { case 0: dma_w<0>(wsrd, wo[0], so, mm_); break; case 1: dma_w<1024>(wsrd, wo[1], so, mm_); break; case 2: dma_w<2048>(wsrd, wo[2], so, mm_); break; default: dma_w<3072>(wsrd, wo[3], so, mm_); }
    }
  };
#pragma unroll
  for (int i = 0; i < 16; ++i) fill1(i >> 3, i & 7);
  float sx = *(const CONSTP float*)asx, sw = *(const CONSTP float*)asw;
  asm volatile("" : "+s"(sx), "+s"(sw));

  i32x4 xf[8][2], xg[8][2];
  i32x4 wf[8][4][2];
  LDSP unsigned char* rd[4];
#pragma unroll
  for (int ct = 0; ct < 4; ++ct)
    rd[ct] = (LDSP unsigned char*)smem + w * RING + (16 * (l >> 4) + ((l & 15) >> 1)) * 64 + ((ct ^ ((l >> 3) & 3)) << 4) + (l & 1) * 8;
  int xr = w * RING + (l & 15) * 128 + (((l >> 4) ^ ((l >> 1) & 7)) << 4);
  int pwa = PART_OFF + w * 4096 + (l & 15) * 256 + (((l >> 4) ^ (l & 7)) << 4);
  int pra = PART_OFF + (w * 4 + (l >> 4)) * 256 + (((l & 15) ^ ((w & 1) * 4 + (l >> 4))) << 4);
  unsigned ooff = ((w * 4 + (l >> 4)) * NS + c0 + (l & 15) * 4) * 4;
  float scale = sx * sw;
  asm volatile("" : "+v"(xr), "+v"(pwa), "+v"(pra), "+v"(ooff), "+v"(scale));
  make_tables();
  auto read_w2 = [&](int kb, int ct, int h) {
    const i32x2 t0 = __builtin_amdgcn_ds_read_tr8_b64_v2i32((LDSP i32x2*)(rd[ct] + (kb & 3) * 8192 + h * 4096));
    const i32x2 t1 = __builtin_amdgcn_ds_read_tr8_b64_v2i32((LDSP i32x2*)(rd[ct] + (kb & 3) * 8192 + h * 4096 + 512));
    wf[kb][ct][h] = i32x4{t0.x, t0.y, t1.x, t1.y};
  };
  auto ldx = [&](int kb, int u, int tog) { return *(LDSP i32x4*)((LDSP unsigned char*)smem + (xr ^ tog ^ (u * 64)) + kb * 2048); };
  auto rx = [&](int i, int tog) { xf[i >> 1][i & 1] = ldx(i >> 1, i & 1, tog); };
  f32x4 acc[4], acc0[4];
  const f32x4 zero = {0.f, 0.f, 0.f, 0.f};
  auto mm = [&](int kb, int ct, const i32x4 (&x)[8][2], f32x4 c) {
    const i32x8 aw8 = __builtin_shufflevector(wf[kb][ct][0], wf[kb][ct][1], 0, 1, 2, 3, 4, 5, 6, 7);
    const i32x8 bx = __builtin_shufflevector(x[kb][0], x[kb][1], 0, 1, 2, 3, 4, 5, 6, 7);
    f32x4 d = __builtin_amdgcn_mfma_scale_f32_16x16x128_f8f6f4(aw8, bx, c, 1, 1, 0, 0, 0, 0);
    asm volatile("" : "+a"(d));
    return d;
  };
  auto mma1 = [&](int kb, int ct) { acc[ct] = mm(kb, ct, xf, kb == 0 ? zero : acc[ct]); };
  auto mma01 = [&](int kb, int i) {
    const int ct = i >> 1;
    if (i & 1) acc[ct] = mm(kb, ct, xf, kb == 0 ? zero : acc[ct]); else acc0[ct] = mm(kb, ct, xg, kb == 0 ? zero : acc0[ct]);
  };
  const i32x4 osrd = make_srd(dout, local ? 0 : NR * MS * NS * 4), lsrd = make_srd(dout, local ? NR * MS * NS * 4 : 0);
  auto pw1 = [&](int ct, int tog, f32x4 v) { *(LDSP f32x4*)((LDSP unsigned char*)smem + (pwa ^ tog ^ ((ct & 1) * 64)) + (ct >> 1) * 128) = v; };
  auto pr1 = [&](int u, int tog) { return *(LDSP f32x4*)((LDSP unsigned char*)smem + (pra ^ tog) + u * 4096); };
  f32x32 res;
  auto keep = [&](int g, f32x4 v) { const int i = (g & 7) * 4; res[i] = v.x; res[i + 1] = v.y; res[i + 2] = v.z; res[i + 3] = v.w; };
  auto send = [&](unsigned orow, f32x4 v) {
    store16(osrd, ooff, orow, __builtin_bit_cast(u32x4, v)); store16_plain(lsrd, ooff, orow, __builtin_bit_cast(u32x4, v));
  };

  const unsigned now0 = lane_s(t_now, 0), now1 = lane_s(t_now, 1);
  STAMP(10);
#pragma unroll
  for (int f = 0; f < 12; ++f) {
    if (f < 11) WAITV(8); else WAITV(4);
    STEP(f);
#pragma unroll
    for (int i = 0; i < 8; ++i) {
      if (f >= 5) mma01(f - 5, i);
      if (f < 10) fill1(f + 2, i);
      else if ((i & 1) == 0) fill1(12, (f - 10) * 4 + (i >> 1));
      if (f < 2) xg[f * 4 + (i >> 1)][i & 1] = ldx(f * 4 + (i >> 1), i & 1, 0);
      else if (f < 4) rx((f - 2) * 8 + i, 16384);
      else read_w2(f - 4, i >> 1, i & 1);
      SB();
    }
    if (f < 4) WAITL(0);
  }
#pragma unroll
  for (int i = 0; i < 8; ++i) {
    mma01(7, i);
    if (i >= 4) { pw1((i - 4) >> 1, (i & 1) * 16384, (i & 1) ? acc[(i - 4) >> 1] : acc0[(i - 4) >> 1]); }
    SB();
  }
  STAMP(2);
  pw1(2, 0, acc0[2]); pw1(2, 16384, acc[2]); pw1(3, 0, acc0[3]); pw1(3, 16384, acc[3]);
  WAITL(0); BAR();
  {
    f32x4 p[8];
#pragma unroll
    for (int u = 0; u < 8; ++u) p[u] = pr1(u & 3, (u >> 2) * 16384);
    const f32x4 s0 = ((p[0] + p[1]) + (p[2] + p[3])) * scale, s1 = ((p[4] + p[5]) + (p[6] + p[7])) * scale;
    send(now0, s0);
    send(now1, s1);
    if (fm & 1) LSTAMP(0);
    if (fm & 2) LSTAMP(1);
    keep(0, s0); keep(1, s1);
  }
  STAMP(3);
  WAITV(4);
#pragma unroll
  for (int i = 0; i < 8; ++i) { rx(i, 0); fill1(13, i); SB(); }
  send(OOB, zero);
#pragma unroll
  for (int i = 0; i < 8; ++i) fill1(14, i);
  WAITL(0); BAR();

#define G(kb, ct, ...) do { mma1(kb, ct); __VA_ARGS__; SB(); } while (0)
  int tog = 0;
  i32x4 xsrd = group_srd(3);
  unsigned orow = OOB;
  const int gpoll = ng > 8 ? 6 : -1;
#pragma clang loop unroll(disable)
  for (int g = 2; g < ng; ++g) {
    asm volatile("" : "+s"(g));
    __builtin_amdgcn_s_waitcnt(0xC07F);
    const int tn = tog ^ 16384;
    f32x4 p0, p1, p2, p3;
    {
      const unsigned sl = ring + 8192 + tn;
      pw1(0, tog, acc[0]); G(0, 0, dmax(xsrd, 0, 512, sl));
      pw1(1, tog, acc[1]); G(0, 1, );
      pw1(2, tog, acc[2]); G(0, 2, dmax(xsrd, 1, 512, sl));
      pw1(3, tog, acc[3]); G(0, 3, WAITV(12); rx(8, tog));
      G(1, 0, dmax(xsrd, 2, 512, sl); rx(9, tog));
      G(1, 1, rx(10, tog); WAITL(3); BAR());
      G(1, 2, dmax(xsrd, 3, 512, sl); p0 = pr1(0, tog); p1 = pr1(1, tog));
      G(1, 3, p2 = pr1(2, tog); p3 = pr1(3, tog));
      G(2, 0, dmax(xsrd, 4, 512, sl); rx(11, tog));
      G(2, 1, rx(12, tog); rx(13, tog));
      G(2, 2, dmax(xsrd, 5, 512, sl); rx(14, tog));
      G(2, 3, rx(15, tog); p0 += p1);
      G(3, 0, dmax(xsrd, 6, 512, sl); p2 += p3);
      G(3, 1, p0 += p2);
      G(3, 2, dmax(xsrd, 7, 512, sl); p0 *= scale);
      G(3, 3, send(orow, p0));
    }
    if (__builtin_expect(g == gpoll, 0)) {
      if (wait_counters(myws, WS_XCNT, call * 8u, call, rank, l)) scale = __builtin_nanf("");
      WAITV(0);
    }
    {
      const unsigned sl = ring + tog;
      G(4, 0, xsrd = group_srd(g + 2));
      G(4, 1, keep(g - 1, p0); asm volatile("" : "+v"(res)));
      G(4, 2, dmax(xsrd, 0, 0, sl));
      G(4, 3, orow = lane_s(t_now, g));
      G(5, 0, dmax(xsrd, 1, 0, sl));
      G(5, 1, WAITV(12); rx(0, tn));
      G(5, 2, dmax(xsrd, 2, 0, sl); rx(1, tn));
      G(5, 3, rx(2, tn));
      G(6, 0, dmax(xsrd, 3, 0, sl); rx(3, tn));
      G(6, 1, rx(4, tn));
      G(6, 2, dmax(xsrd, 4, 0, sl); rx(5, tn));
      G(6, 3, rx(6, tn));
      G(7, 0, dmax(xsrd, 5, 0, sl); rx(7, tn));
      G(7, 1, dmax(xsrd, 6, 0, sl));
      G(7, 2, dmax(xsrd, 7, 0, sl));
      G(7, 3, );
    }
    tog = tn;
  }
  pw1(0, tog, acc[0]); pw1(1, tog, acc[1]); pw1(2, tog, acc[2]); pw1(3, tog, acc[3]);
  WAITL(0); BAR();
  {
    const f32x4 s = ((pr1(0, tog) + pr1(1, tog)) + (pr1(2, tog) + pr1(3, tog))) * scale;
    STAMP(11);
    send(lane_s(t_row, ng - 1), s);
    LSTAMP(ng - 1);
  }
#pragma unroll
  for (int g = 0; g < 7; ++g) {
    if (!((fm >> g) & 1) && g < ng - 1) {
      if (win == 1) WAITV(0); else if (win == 2) WAITV(1); else if (win == 3) WAITV(2);
      store16(osrd, ooff, lane_s(t_row, g), __builtin_bit_cast(u32x4, f32x4{res[4 * g], res[4 * g + 1], res[4 * g + 2], res[4 * g + 3]}));
      LSTAMP(g);
    }
  }
  STAMP(4);
  WAITV(0);
  BAR();
  STAMP(5);
  if (threadIdx.x == 0) {
    if (!local) __hip_atomic_fetch_add((gu32*)(dws + WS_OCNT + rank * 128), 1u, __ATOMIC_RELAXED, __HIP_MEMORY_SCOPE_SYSTEM);
    __hip_atomic_fetch_add((gu32*)(myws + WS_DONE), 1u, __ATOMIC_RELAXED, __HIP_MEMORY_SCOPE_AGENT);
  }
  if (local && j == 0 && w == 2) {
    if (wait_counters(myws, WS_OCNT, call * 32u, call, rank, l) && l == 0)
      ((gf32*)dout)[(size_t)(rank * MS + r0) * NS + c0] = __builtin_nanf("");
  }
}

extern "C" void kernel_launch(int rank, int nranks, ncclComm_t comm, hipStream_t stream, void* const* d_in, const int* in_sizes, int n_in,
                              void* d_out, int out_size, const int* in_dtypes, int out_dtype, void* const* d_out_all, void* const* d_ws_all, size_t ws_bytes) {
  static bool init = false;
  if (!init) { (void)hipFuncSetAttribute((const void*)gemm_a2a, hipFuncAttributeMaxDynamicSharedMemorySize, LDS_BYTES); init = true; }
  Args a;
  for (int i = 0; i < NR; ++i) { int p = (rank + 1 + i) & 7; a.peer[i].out = d_out_all[p]; a.peer[i].ws = d_ws_all[p]; }
  a.x = d_in[0]; a.w = d_in[1]; a.sx = d_in[2]; a.sw = d_in[3];
  a.rank = rank; a.myws = d_ws_all[rank];
  auto env = [](const char* n, int d) { return d; };
  a.cfg = env("NRG", 4) | (env("PF", 1) << 4) | (env("PWIN", 2) << 8); a.d12 = env("D1", 0) | (env("D2", 350) << 16); a.d3l = env("D3", 500) | (env("DLOC", 100) << 16);
  a.fmask = env("F0", 255) | (env("F1", 255) << 8) | (env("F2", 255) << 16) | (env("F3", 0) << 24); a.win = env("WIN", 2) | (env("DL0", 200) << 4);
  gemm_a2a<<<256, 256, LDS_BYTES, stream>>>(a);
}
```

```cpp
#include <hip/hip_runtime.h>
#include <rccl/rccl.h>
#include <cstdio>
#include <cstdlib>

typedef unsigned u32x4 __attribute__((ext_vector_type(4)));
typedef int i32x2 __attribute__((ext_vector_type(2)));
typedef int i32x4 __attribute__((ext_vector_type(4)));
typedef int i32x8 __attribute__((ext_vector_type(8)));
typedef int i32x16 __attribute__((ext_vector_type(16)));
typedef float f32x4 __attribute__((ext_vector_type(4)));
typedef float f32x32 __attribute__((ext_vector_type(32)));
typedef unsigned long long u64;
#define LDSP __attribute__((address_space(3)))
#define CONSTP __attribute__((address_space(4)))
typedef __attribute__((address_space(1))) unsigned gu32;
typedef __attribute__((address_space(1))) float gf32;
typedef __attribute__((address_space(1))) u64 gu64;

constexpr int NR = 8, MS = 512, KD = 4096, ND = 4096, NS = 512;
constexpr int RING = 32768;
constexpr int PART_OFF = 4 * RING;
constexpr int LDS_BYTES = PART_OFF + 2 * 16384;
constexpr u64 DEADLINE = 20000000ull;
constexpr unsigned WS_DONE = 0, WS_TMO = 128, WS_OCNT = 4096, WS_XCNT = 8192, WS_XS = 65536;
constexpr unsigned XS_SRC = 64 * KD, XS_PAR = NR * XS_SRC;
constexpr unsigned OOB = 0x40000000u;
#define LATE 1
#define ROT 0
#define STAMP(i)
#define LSTAMP(i)
#define STEP(i)

struct Args {
  const void* x; const void* w; void* myws; const void* sx; const void* sw;
  int rank, cfg, d12, d3l, fmask, win;
  struct { void* out; void* ws; } peer[NR];
};

#define WAITV(n) asm volatile("s_waitcnt vmcnt(%0)" ::"n"(n) : "memory")
#define WAITL(n) asm volatile("s_waitcnt lgkmcnt(%0)" ::"n"(n) : "memory")
#define SB() __builtin_amdgcn_sched_barrier(0)
#define BAR() do { __builtin_amdgcn_s_barrier(); asm volatile("" ::: "memory"); } while (0)

template <int OFF>
__device__ __forceinline__ void dma_w(i32x4 srd, unsigned voff, unsigned soff, unsigned lds) {
  asm volatile("s_mov_b32 m0, %4\n\ts_nop 0\n\tbuffer_load_dwordx4 %0, %1, %2 offen offset:%3 lds" ::"v"(voff), "s"(srd), "s"(soff), "n"(OFF), "s"(lds) : "memory");
}
template <int OFF>
__device__ __forceinline__ void dma_x(i32x4 srd, unsigned voff, unsigned soff, unsigned lds) {
  asm volatile("s_mov_b32 m0, %4\n\ts_nop 0\n\tbuffer_load_dwordx4 %0, %1, %2 offen offset:%3 sc0 sc1 lds" ::"v"(voff), "s"(srd), "s"(soff), "n"(OFF), "s"(lds) : "memory");
}
__device__ __forceinline__ void dma_touch(i32x4 srd, unsigned voff, unsigned soff, unsigned lds) {
  asm volatile("s_mov_b32 m0, %3\n\ts_nop 0\n\tbuffer_load_dword %0, %1, %2 offen lds" ::"v"(voff), "s"(srd), "s"(soff), "s"(lds) : "memory");
}
__device__ __forceinline__ i32x4 make_srd(u64 a, unsigned bytes) {
  i32x4 r; r.x = (int)(unsigned)a; r.y = (int)((unsigned)(a >> 32) & 0xffffu); r.z = (int)bytes; r.w = 0x00020000; return r;
}
__device__ __forceinline__ void store16(i32x4 srd, unsigned voff, unsigned soff, u32x4 v) {
  asm volatile("buffer_store_dwordx4 %0, %1, %2, %3 offen sc0 sc1\n\ts_nop 1" ::"v"(v), "v"(voff), "s"(srd), "s"(soff) : "memory");
}
__device__ __forceinline__ void store16_plain(i32x4 srd, unsigned voff, unsigned soff, u32x4 v) {
  asm volatile("buffer_store_dwordx4 %0, %1, %2, %3 offen\n\ts_nop 1" ::"v"(v), "v"(voff), "s"(srd), "s"(soff) : "memory");
}
__device__ __forceinline__ bool wait_counters(u64 myws, unsigned off, unsigned target, unsigned call, int rank, int l) {
  bool bad = false;
  if (l < NR && l != rank) {
    gu32* f = (gu32*)(myws + off + l * 128);
    gu32* tmo = (gu32*)(myws + WS_TMO);
    const u64 t0 = __builtin_amdgcn_s_memrealtime();
    while (__hip_atomic_load(f, __ATOMIC_RELAXED, __HIP_MEMORY_SCOPE_SYSTEM) < target) {
      if (__hip_atomic_load(tmo, __ATOMIC_RELAXED, __HIP_MEMORY_SCOPE_AGENT) == call || __builtin_amdgcn_s_memrealtime() - t0 > DEADLINE) {
        __hip_atomic_store(tmo, call, __ATOMIC_RELAXED, __HIP_MEMORY_SCOPE_AGENT);
        bad = true;
        break;
      }
      __builtin_amdgcn_s_sleep(1);
    }
  }
  return __builtin_amdgcn_ballot_w64(bad) != 0;
}

__global__ __launch_bounds__(256, 1) void gemm_a2a(Args args) {
  extern __shared__ __attribute__((aligned(16))) unsigned char smem[];
  STAMP(0);
  const u64 tstart = __builtin_amdgcn_s_memrealtime();
  const int b = blockIdx.x, xcd = b & 7, j = b >> 3, blk = j >> 3, ph = j & 1, qr = (j >> 1) & 3;
  const CONSTP unsigned char* kp = (const CONSTP unsigned char*)__builtin_amdgcn_kernarg_segment_ptr();
  i32x16 hv = *(const CONSTP i32x16*)kp;
  const int pi = (xcd + ROT * blk) & 7;
  i32x4 pv = *(const CONSTP i32x4*)(kp + 64 + pi * 16);
  asm volatile("" : "+s"(hv), "+s"(pv));
  auto mk64 = [](int lo, int hi) { return (unsigned)lo | ((u64)(unsigned)hi << 32); };
  const u64 ax = mk64(hv[0], hv[1]), aw = mk64(hv[2], hv[3]), myws = mk64(hv[4], hv[5]), asx = mk64(hv[6], hv[7]), asw = mk64(hv[8], hv[9]);
  const int rank = hv[10], nR = hv[11] & 7, pf = (hv[11] >> 4) & 1, pwin = (hv[11] >> 8) & 15, lsc = (hv[11] >> 12) & 3, fastp = (hv[11] >> 14) & 1, dpf1 = (hv[11] >> 16) & 255, d12 = hv[12], d3l = hv[13], fmask = hv[14], win = hv[15] & 15, dl0 = hv[15] >> 4;
  const u64 dout = mk64(pv.x, pv.y), dws = mk64(pv.z, pv.w);
  const int dest = (rank + 1 + pi) & 7;
  const bool local = pi == 7;
  const int c0 = blk * 128 + ph * 64, n0 = dest * NS + c0, r0 = qr * 128;
  const int w = __builtin_amdgcn_readfirstlane(threadIdx.x >> 6), l = threadIdx.x & 63;
  const unsigned lds0 = (unsigned)(size_t)(LDSP unsigned char*)smem;
  const unsigned ring = lds0 + w * RING;
  const i32x4 wsrd = make_srd(aw, KD * ND);
  unsigned call = 0;

  {
    const i32x4 xall = make_srd(ax, MS * KD);
    auto until = [&](int ticks) { while ((int)(unsigned)(__builtin_amdgcn_s_memrealtime() - tstart) < ticks) __builtin_amdgcn_s_sleep(1); };
    const unsigned pfs = pf ? 0u : OOB, scr = lds0 + PART_OFF + w * 4096;
    auto pf_w = [&](int pnl, int c) {
      const int k = ((c >> 1) & 3) * 1024 + (c >> 3) * 128 + (c & 1) * 64 + l;
      dma_touch(wsrd, (unsigned)k * ND + ((rank + 1 + xcd + ROT * pnl) & 7) * NS + pnl * 128, pfs, scr);
    };
    auto pf_x = [&](int g, int c) {
      const int row = (c >> 3) * 128 + g * 16 + (c & 7) * 2 + (l >> 5);
      dma_touch(xall, (unsigned)row * KD + (l & 31) * 128, pfs, scr);
    };
    const int pw = (j & 7) * 4 + w;
    const int tgo = local ? dl0 + (d3l >> 16) * blk : blk == 0 ? 0 : blk == 1 ? (d12 & 0xffff) : blk == 2 ? (d12 >> 16) : (d3l & 0xffff);
    if (local) {
      if (blk >= 2) { until(dl0 - 100); pf_x(blk - 2, pw); }
    } else if (blk < 2) {
      pf_x(blk, pw);
    } else if (blk == 3) {
      unsigned done;
      u32x4 v[8];
      const unsigned poff = (pw >> 1) * KD + (pw & 1) * 2048 + l * 16;
      auto ld2 = [&](int i) {
        asm volatile("buffer_load_dwordx4 %0, %2, %3, %4 offen\n\tbuffer_load_dwordx4 %1, %2, %3, %4 offen offset:1024"
                     : "=&v"(v[2 * i]), "=&v"(v[2 * i + 1]) : "v"(poff), "s"(xall), "s"(i < nR ? (unsigned)((3 - i) * 128 + 112) * KD : OOB) : "memory");
      };
      asm volatile("global_load_dword %0, %1, %2 sc1" : "=&v"(done) : "v"(WS_DONE), "s"(myws) : "memory");
      ld2(0);
      asm volatile("s_waitcnt vmcnt(0)" : "+v"(done), "+v"(v[0]), "+v"(v[1])::"memory");
      ld2(1); ld2(2); ld2(3);
      call = __builtin_amdgcn_readfirstlane(done) / 256u + 1u;
      const i32x4 psrd = make_srd(dws + WS_XS + (call & 1) * XS_PAR + rank * XS_SRC, nR * 16 * KD);
#pragma unroll
      for (int i = 0; i < 4; ++i) {
        if (i) asm volatile("s_waitcnt vmcnt(6)" : "+v"(v[2 * i]), "+v"(v[2 * i + 1])::"memory");
        if (i && pwin == 2) WAITV(1);
        if (i && pwin == 3) WAITV(2);
        store16(psrd, poff, i * 16 * KD, v[2 * i]);
        if (i && pwin == 2) WAITV(1);
        if (i && pwin == 3) WAITV(2);
        store16(psrd, poff + 1024, i * 16 * KD, v[2 * i + 1]);
        if (i == 0) STAMP(8);
      }
      WAITV(0);
      BAR();
      STAMP(9);
      if (threadIdx.x == 0 && nR)
        __hip_atomic_fetch_add((gu32*)(dws + WS_XCNT + rank * 128), 1u, __ATOMIC_RELAXED, __HIP_MEMORY_SCOPE_SYSTEM);
    }
    if (blk == 2 && !local) {
      until(dpf1);
#pragma unroll
      for (int q = 0; q < 4; ++q) pf_w(pw >> 4, q * 16 + (pw & 15));
      pf_x(2, pw);
    }
    if (blk >= 2 || local) { until(tgo - 100); pf_w(blk, pw); pf_w(blk, 32 + pw); }
    if (tgo) until(tgo);
  }
  STAMP(1);

  unsigned xo[4], wo[4];
#pragma unroll
  for (int i = 0; i < 4; ++i) {
    xo[i] = 4096 + w * 1024 + ((l >> 3) + (i & 1) * 8) * KD + (((l & 7) ^ ((l >> 4) & 3) ^ ((i & 1) << 2)) << 4) + (i >> 1) * 128 - i * 1024;
    wo[i] = (w * 1024 + (l >> 2) + i * 16) * ND + (((l & 3) ^ ((l >> 4) & 1) ^ ((i & 1) << 1)) << 4) + n0 - i * 1024;
  }
  auto dmax = [&](i32x4 srd, int i, unsigned so, unsigned slot) {
    const unsigned s = so + (i >> 2) * 256, m = slot + (i >> 2) * 4096;
    switch (i & 3) { case 0: dma_x<0>(srd, xo[0], s, m); break; case 1: dma_x<1024>(srd, xo[1], s, m); break; case 2: dma_x<2048>(srd, xo[2], s, m); break; default: dma_x<3072>(srd, xo[3], s, m); }
  };
  const int nown = (!local && qr >= 4 - nR) ? 7 : 8;
  const int ng = local ? 8 + (7 * nR + 3 - qr) / 4 : nown;
  const u64 xown = ax + (unsigned)r0 * KD - 4096;
  const unsigned fm = local ? 0xffffu : (unsigned)(fmask >> (blk * 8)) & 0xffu;
  unsigned t_lo, t_hi, t_rec, t_row, t_now;
  auto make_tables = [&](int g) {
    const int f = qr + 4 * (g - 8), i = (f * 37) >> 8, src = (rank + 1 + f - 7 * i) & 7;
    const u64 xfor = myws + (WS_XS - 4096) + (call & 1) * XS_PAR;
    const u64 a = g < 8 ? xown + (unsigned)g * 16 * KD : xfor + (unsigned)(src * XS_SRC + i * 16 * KD);
    t_lo = (unsigned)a; t_hi = (unsigned)(a >> 32) & 0xffffu; t_rec = g < ng ? 17 * KD : 0;
    t_row = (unsigned)(g < 8 ? rank * MS + r0 + g * 16 : src * MS + (3 - i) * 128 + 112) * (NS * 4);
    t_now = ((fm >> (g & 15)) & 1) ? t_row : OOB;
    asm volatile("" : "+v"(t_lo), "+v"(t_hi), "+v"(t_rec), "+v"(t_row), "+v"(t_now));
  };
  auto group_srd = [&](int g) {
    i32x4 r = {__builtin_amdgcn_readlane((int)t_lo, g), __builtin_amdgcn_readlane((int)t_hi, g), __builtin_amdgcn_readlane((int)t_rec, g), 0x00020000};
    asm volatile("s_nop 4" : "+s"(r));
    return r;
  };
  auto lane_s = [&](unsigned t, int g) { unsigned r = __builtin_amdgcn_readlane((int)t, g); asm volatile("s_nop 4" : "+s"(r)); return r; };
  const i32x4 x03 = make_srd(xown, 65 * KD);
  auto fill1 = [&](int f, int i) {
    const unsigned m = ring + (f & 3) * 8192;
    if (f < 4 || f >= 12) {
      const int h = f < 4 ? f : f - 8;
      dmax(x03, i, (h >> 1) * 16 * KD + (h & 1) * 512, m);
    } else {
      const unsigned so = (unsigned)((f - 4) * 128 + (i >> 2) * 64) * ND, mm_ = m + (i >> 2) * 4096;
      switch (i & 3) { case 0: dma_w<0>(wsrd, wo[0], so, mm_); break; case 1: dma_w<1024>(wsrd, wo[1], so, mm_); break; case 2: dma_w<2048>(wsrd, wo[2], so, mm_); break; default: dma_w<3072>(wsrd, wo[3], so, mm_); }
    }
  };
  float sx, sw;
  unsigned done;
  asm volatile("global_load_dword %0, %3, %4\n\tglobal_load_dword %1, %3, %5\n\tbuffer_load_dword %2, %3, %6, 0 offen sc1"
               : "=&v"(sx), "=&v"(sw), "=&v"(done) : "v"(0), "s"(asx), "s"(asw), "s"(make_srd(myws + WS_DONE, local ? 4 : 0)) : "memory");
#pragma unroll
  for (int i = 0; i < 16; ++i) fill1(i >> 3, i & 7);

  i32x4 xf[8][2], xg[8][2];
  i32x4 wf[8][4][2];
  LDSP unsigned char* rd[4];
#pragma unroll
  for (int ct = 0; ct < 4; ++ct)
    rd[ct] = (LDSP unsigned char*)smem + w * RING + (16 * (l >> 4) + ((l & 15) >> 1)) * 64 + ((ct ^ ((l >> 3) & 3)) << 4) + (l & 1) * 8;
  int xr = w * RING + (l & 15) * 128 + (((l >> 4) ^ ((l >> 1) & 7)) << 4);
  int pwa, pra;
  unsigned ooff;
  float scale;
  unsigned now0, now1;
  auto read_w2 = [&](int kb, int ct, int h) {
    const i32x2 t0 = __builtin_amdgcn_ds_read_tr8_b64_v2i32((LDSP i32x2*)(rd[ct] + (kb & 3) * 8192 + h * 4096));
    const i32x2 t1 = __builtin_amdgcn_ds_read_tr8_b64_v2i32((LDSP i32x2*)(rd[ct] + (kb & 3) * 8192 + h * 4096 + 512));
    wf[kb][ct][h] = i32x4{t0.x, t0.y, t1.x, t1.y};
  };
  auto ldx = [&](int kb, int u, int tog) { return *(LDSP i32x4*)((LDSP unsigned char*)smem + (xr ^ tog ^ (u * 64)) + kb * 2048); };
  auto rx = [&](int i, int tog) { xf[i >> 1][i & 1] = ldx(i >> 1, i & 1, tog); };
  f32x4 acc[4], acc0[4];
  const f32x4 zero = {0.f, 0.f, 0.f, 0.f};
  auto mm = [&](int kb, int ct, const i32x4 (&x)[8][2], f32x4 c) {
    const i32x8 aw8 = __builtin_shufflevector(wf[kb][ct][0], wf[kb][ct][1], 0, 1, 2, 3, 4, 5, 6, 7);
    const i32x8 bx = __builtin_shufflevector(x[kb][0], x[kb][1], 0, 1, 2, 3, 4, 5, 6, 7);
    f32x4 d = __builtin_amdgcn_mfma_scale_f32_16x16x128_f8f6f4(aw8, bx, c, 1, 1, 0, 0, 0, 0);
    asm volatile("" : "+a"(d));
    return d;
  };
  auto mma1 = [&](int kb, int ct) { acc[ct] = mm(kb, ct, xf, kb == 0 ? zero : acc[ct]); };
  auto mma01 = [&](int kb, int i) {
    const int ct = i >> 1;
    if (i & 1) acc[ct] = mm(kb, ct, xf, kb == 0 ? zero : acc[ct]); else acc0[ct] = mm(kb, ct, xg, kb == 0 ? zero : acc0[ct]);
  };
  const i32x4 osrd = make_srd(dout, NR * MS * NS * 4);
  auto pw1 = [&](int ct, int tog, f32x4 v) { *(LDSP f32x4*)((LDSP unsigned char*)smem + (pwa ^ tog ^ ((ct & 1) * 64)) + (ct >> 1) * 128) = v; };
  auto pr1 = [&](int u, int tog) { return *(LDSP f32x4*)((LDSP unsigned char*)smem + (pra ^ tog) + u * 4096); };
  f32x32 res;
  auto keep = [&](int g, f32x4 v) { const int i = (g & 7) * 4; res[i] = v.x; res[i + 1] = v.y; res[i + 2] = v.z; res[i + 3] = v.w; };
  auto send = [&](unsigned orow, f32x4 v) {
    store16(osrd, ooff, orow, __builtin_bit_cast(u32x4, v));
  };

  STAMP(10);
#pragma unroll
  for (int f = 0; f < 11; ++f) {
    WAITV(8);
    STEP(f);
#pragma unroll
    for (int i = 0; i < 8; ++i) {
      if (f >= 5) mma01(f - 5, i);
      if (f < 10) fill1(f + 2, i);
      else if ((i & 1) == 0) fill1(12, (f - 10) * 4 + (i >> 1));
      if (f < 2) xg[f * 4 + (i >> 1)][i & 1] = ldx(f * 4 + (i >> 1), i & 1, 0);
      else if (f < 4) rx((f - 2) * 8 + i, 16384);
      else read_w2(f - 4, i >> 1, i & 1);
      SB();
    }
    if (f == 4) {
      int m = l;
      asm volatile("" : "+v"(m), "+v"(sx), "+v"(sw), "+v"(done));
      call = __builtin_amdgcn_readfirstlane(done) / 256u + 1u;
      scale = sx * sw;
      pwa = PART_OFF + w * 4096 + (m & 15) * 256 + (((m >> 4) ^ (m & 7)) << 4);
      pra = PART_OFF + (w * 4 + (m >> 4)) * 256 + (((m & 15) ^ ((w & 1) * 4 + (m >> 4))) << 4);
      ooff = ((w * 4 + (m >> 4)) * NS + c0 + (m & 15) * 4) * 4;
      asm volatile("" : "+v"(pwa), "+v"(pra), "+v"(ooff), "+v"(scale));
      make_tables(m);
      now0 = lane_s(t_now, 0); now1 = lane_s(t_now, 1);
      SB();
    }
    if (f < 4) WAITL(0);
  }
#pragma unroll
  for (int i = 0; i < 8; ++i) {
    mma01(6, i);
    if (i < 4) fill1(12, 4 + i);
    if (i == 3) { WAITV(8); STEP(11); }
    if (i >= 4) { read_w2(7, i - 4, 0); read_w2(7, i - 4, 1); }
    SB();
  }
#pragma unroll
  for (int i = 0; i < 8; ++i) {
    mma01(7, i);
    if (i >= 4) { pw1((i - 4) >> 1, (i & 1) * 16384, (i & 1) ? acc[(i - 4) >> 1] : acc0[(i - 4) >> 1]); }
    SB();
  }
  STAMP(2);
  pw1(2, 0, acc0[2]); pw1(2, 16384, acc[2]); pw1(3, 0, acc0[3]); pw1(3, 16384, acc[3]);
  WAITL(0); BAR();
  {
    f32x4 p[8];
#pragma unroll
    for (int u = 0; u < 8; ++u) p[u] = pr1(u & 3, (u >> 2) * 16384);
    const f32x4 s0 = ((p[0] + p[1]) + (p[2] + p[3])) * scale, s1 = ((p[4] + p[5]) + (p[6] + p[7])) * scale;
    send(now0, s0);
    send(now1, s1);
    if (fm & 1) LSTAMP(0);
    if (fm & 2) LSTAMP(1);
    keep(0, s0); keep(1, s1);
  }
  STAMP(3);
  WAITV(2);
#pragma unroll
  for (int i = 0; i < 8; ++i) { rx(i, 0); fill1(13, i); SB(); }
#pragma unroll
  for (int i = 0; i < 8; ++i) fill1(14, i);
  send(OOB, zero);
  WAITL(0); BAR();

#define G(kb, ct, ...) do { mma1(kb, ct); __VA_ARGS__; SB(); } while (0)
  int tog = 0;
  i32x4 xsrd = group_srd(3);
  unsigned orow = OOB;
  const int gpoll = ng > 8 ? 6 : -1;
#pragma clang loop unroll(disable)
  for (int g = 2; g < ng; ++g) {
    asm volatile("" : "+s"(g));
    __builtin_amdgcn_s_waitcnt(0xC07F);
    const int tn = tog ^ 16384;
    f32x4 p0, p1, p2, p3;
    const f32x4 c0_ = acc[0], c1_ = acc[1], c2_ = acc[2], c3_ = acc[3];
    {
      const unsigned sl = ring + 8192 + tn;
      G(0, 0, pw1(0, tog, c0_));
      G(0, 1, dmax(xsrd, 0, 512, sl));
      G(0, 2, WAITV(10); rx(8, tog));
      G(0, 3, dmax(xsrd, 1, 512, sl); rx(9, tog));
      G(1, 0, pw1(1, tog, c1_));
      G(1, 1, dmax(xsrd, 2, 512, sl); rx(10, tog));
      G(1, 2, rx(11, tog));
      G(1, 3, dmax(xsrd, 3, 512, sl); rx(12, tog));
      G(2, 0, pw1(2, tog, c2_));
      G(2, 1, dmax(xsrd, 4, 512, sl); rx(13, tog));
      G(2, 2, rx(14, tog));
      G(2, 3, dmax(xsrd, 5, 512, sl); rx(15, tog));
      G(3, 0, pw1(3, tog, c3_));
      G(3, 1, dmax(xsrd, 6, 512, sl));
      G(3, 2, );
      G(3, 3, dmax(xsrd, 7, 512, sl));
    }
    if (__builtin_expect(g == gpoll, 0)) {
      if (wait_counters(myws, WS_XCNT, call * 8u, call, rank, l)) scale = __builtin_nanf("");
      WAITV(0);
    }
    {
      const unsigned sl = ring + tog;
      G(4, 0, xsrd = group_srd(g + 2); WAITL(0); BAR());
      G(4, 1, dmax(xsrd, 0, 0, sl); p0 = pr1(0, tog));
      G(4, 2, p1 = pr1(1, tog); p2 = pr1(2, tog));
      G(4, 3, dmax(xsrd, 1, 0, sl); p3 = pr1(3, tog));
      G(5, 0, WAITV(11); rx(0, tn));
      G(5, 1, dmax(xsrd, 2, 0, sl); rx(1, tn));
      G(5, 2, rx(2, tn); p0 += p1);
      G(5, 3, dmax(xsrd, 3, 0, sl); rx(3, tn));
      G(6, 0, rx(4, tn); p2 += p3);
      G(6, 1, dmax(xsrd, 4, 0, sl); rx(5, tn));
      G(6, 2, rx(6, tn); p0 += p2);
      G(6, 3, dmax(xsrd, 5, 0, sl); rx(7, tn));
      G(7, 0, p0 *= scale);
      G(7, 1, dmax(xsrd, 6, 0, sl));
      G(7, 2, keep(g - 1, p0); asm volatile("" : "+v"(res)));
      G(7, 3, dmax(xsrd, 7, 0, sl); send(orow, p0));
      orow = lane_s(t_now, g);
    }
    tog = tn;
  }
  pw1(0, tog, acc[0]); pw1(1, tog, acc[1]); pw1(2, tog, acc[2]); pw1(3, tog, acc[3]);
  WAITL(0); BAR();
  {
    const f32x4 s = ((pr1(0, tog) + pr1(1, tog)) + (pr1(2, tog) + pr1(3, tog))) * scale;
    STAMP(11);
    send(lane_s(t_row, ng - 1), s);
    LSTAMP(ng - 1);
  }
#pragma unroll
  for (int g = 0; g < 7; ++g) {
    if (!((fm >> g) & 1) && g < ng - 1) {
      if (win == 1) WAITV(0); else if (win == 2) WAITV(1); else if (win == 3) WAITV(2);
      store16(osrd, ooff, lane_s(t_row, g), __builtin_bit_cast(u32x4, f32x4{res[4 * g], res[4 * g + 1], res[4 * g + 2], res[4 * g + 3]}));
      LSTAMP(g);
    }
  }
  STAMP(4);
  WAITV(0);
  BAR();
  STAMP(5);
  if (threadIdx.x == 0) {
    if (!local) __hip_atomic_fetch_add((gu32*)(dws + WS_OCNT + rank * 128), 1u, __ATOMIC_RELAXED, __HIP_MEMORY_SCOPE_SYSTEM);
    __hip_atomic_fetch_add((gu32*)(myws + WS_DONE), 1u, __ATOMIC_RELAXED, __HIP_MEMORY_SCOPE_AGENT);
  }
  if (local && j == 0 && w == 2 && fastp) {
    const unsigned off = l < 8 ? WS_OCNT + l * 128 : WS_TMO;
    const unsigned target = call * 32u;
    const u64 t0 = __builtin_amdgcn_s_memrealtime();
    const u64 mine = 0xffull & ~(1ull << rank);
    bool bad = false;
    if (l < 9) {
      unsigned v0, v1, v2, v3;
#define PL(v) asm volatile("global_load_dword %0, %1, %2 sc0 sc1" : "=&v"(v) : "v"(off), "s"(myws) : "memory")
#define PC(v) asm volatile("s_waitcnt vmcnt(3)" : "+v"(v)::"memory"); \
      if ((__builtin_amdgcn_ballot_w64(v >= target) & mine) == mine) break; \
      if (__builtin_amdgcn_readlane((int)v, 8) == (int)call) { bad = true; break; } \
      PL(v); __builtin_amdgcn_s_sleep(2);
      PL(v0); __builtin_amdgcn_s_sleep(2); PL(v1); __builtin_amdgcn_s_sleep(2); PL(v2); __builtin_amdgcn_s_sleep(2); PL(v3);
      for (;;) {
        PC(v0) PC(v1) PC(v2) PC(v3)
        if (__builtin_amdgcn_s_memrealtime() - t0 > DEADLINE) { bad = true; break; }
      }
      if (bad)
      asm volatile("s_waitcnt vmcnt(0)" : "+v"(v0), "+v"(v1), "+v"(v2), "+v"(v3)::"memory");
    }
    if (bad && l == 0) {
      __hip_atomic_store((gu32*)(myws + WS_TMO), call, __ATOMIC_RELAXED, __HIP_MEMORY_SCOPE_AGENT);
      ((gf32*)dout)[(size_t)(rank * MS + r0) * NS + c0] = __builtin_nanf("");
    }
  } else if (local && j == 0 && w == 2) {
    if (wait_counters(myws, WS_OCNT, call * 32u, call, rank, l) && l == 0)
      ((gf32*)dout)[(size_t)(rank * MS + r0) * NS + c0] = __builtin_nanf("");
  }
}

extern "C" void kernel_launch(int rank, int nranks, ncclComm_t comm, hipStream_t stream, void* const* d_in, const int* in_sizes, int n_in,
                              void* d_out, int out_size, const int* in_dtypes, int out_dtype, void* const* d_out_all, void* const* d_ws_all, size_t ws_bytes) {
  static bool init = false;
  if (!init) { (void)hipFuncSetAttribute((const void*)gemm_a2a, hipFuncAttributeMaxDynamicSharedMemorySize, LDS_BYTES); init = true; }
  Args a;
  for (int i = 0; i < NR; ++i) { int p = (rank + 1 + i) & 7; a.peer[i].out = d_out_all[p]; a.peer[i].ws = d_ws_all[p]; }
  a.x = d_in[0]; a.w = d_in[1]; a.sx = d_in[2]; a.sw = d_in[3];
  a.rank = rank; a.myws = d_ws_all[rank];
  auto env = [](const char* n, int d) { return d; };
  a.cfg = env("NRG", 4) | (env("PF", 1) << 4) | (env("PWIN", 2) << 8) | (env("LSC", 1) << 12) | (env("FASTP", 1) << 14) | (env("DPF1", 70) << 16); a.d12 = env("D1", 0) | (env("D2", 350) << 16); a.d3l = env("D3", 500) | (env("DLOC", 100) << 16);
  a.fmask = env("F0", 255) | (env("F1", 255) << 8) | (env("F2", 255) << 16) | (env("F3", 0) << 24); a.win = env("WIN", 2) | (env("DL0", 150) << 4);
  gemm_a2a<<<256, 256, LDS_BYTES, stream>>>(a);
}
```

```cpp
#include <hip/hip_runtime.h>
#include <rccl/rccl.h>
#include <cstdio>
#include <cstdlib>

typedef unsigned u32x4 __attribute__((ext_vector_type(4)));
typedef int i32x2 __attribute__((ext_vector_type(2)));
typedef int i32x4 __attribute__((ext_vector_type(4)));
typedef int i32x8 __attribute__((ext_vector_type(8)));
typedef int i32x16 __attribute__((ext_vector_type(16)));
typedef float f32x4 __attribute__((ext_vector_type(4)));
typedef unsigned long long u64;
#define LDSP __attribute__((address_space(3)))
#define CONSTP __attribute__((address_space(4)))
typedef __attribute__((address_space(1))) unsigned gu32;
typedef __attribute__((address_space(1))) float gf32;
typedef __attribute__((address_space(1))) u64 gu64;

constexpr int NR = 8, MS = 512, KD = 4096, ND = 4096, NS = 512;
constexpr int STG = 32768;
constexpr int LDS_BYTES = 4 * STG + 1024;
constexpr u64 DEADLINE = 20000000ull;
constexpr unsigned WS_TMO = 128, WS_OCNT = 4096, WS_XCNT = 8192, WS_TAIL = 12288, WS_CALL = 65536, WS_PKF = 131072, WS_XS = 524288;
constexpr unsigned XS_SRC = 64 * KD, XS_PAR = NR * XS_SRC;
constexpr unsigned WS_PK = 8u << 20, PK_RUN = 16384, PK_WG = 2 * PK_RUN, PK_SRC = 32 * PK_WG, PK_PAR = NR * PK_SRC;
constexpr unsigned OOB = 0x40000000u;
#define TLEFT 6
#define WIN 2
#define WIN1 3
#define PSLEEP 1
#define TWIN 8
#define CODE_BYTES 30720
#define STAMP(i)
#define LSTAMP(i)

struct Args {
  const void* x; const void* w; void* myws; const void* sx; const void* sw;
  int rank, cfg, hold, nraw, npk, pad0;
  struct { void* out; void* ws; } peer[NR];
  void* myout; int pad1, pad2;
  unsigned unp[32];
  unsigned job[64];
};

#define WAITV(n) asm volatile("s_waitcnt vmcnt(%0)" ::"n"(n) : "memory")
#define WAITL(n) asm volatile("s_waitcnt lgkmcnt(%0)" ::"n"(n) : "memory")
#define SB() __builtin_amdgcn_sched_barrier(0)
#define BAR() do { __builtin_amdgcn_s_barrier(); asm volatile("" ::: "memory"); } while (0)

template <int LOFF, int OFF>
__device__ __forceinline__ void dma_w(i32x4 srd, unsigned voff, unsigned soff, unsigned lds) {
  asm volatile("s_add_u32 m0, %4, %5\n\ts_nop 0\n\tbuffer_load_dwordx4 %0, %1, %2 offen offset:%3 lds" ::"v"(voff), "s"(srd), "s"(soff), "n"(OFF), "s"(lds), "n"(LOFF) : "memory", "scc");
}
template <int LOFF>
__device__ __forceinline__ void dma_x(i32x4 srd, unsigned voff, unsigned soff, unsigned lds) {
  asm volatile("s_add_u32 m0, %3, %4\n\ts_nop 0\n\tbuffer_load_dwordx4 %0, %1, %2 offen sc0 sc1 lds" ::"v"(voff), "s"(srd), "s"(soff), "s"(lds), "n"(LOFF) : "memory", "scc");
}
template <int LOFF>
__device__ __forceinline__ void dma_xs(i32x4 srd, unsigned voff, unsigned soff, unsigned lds, int skip) {
  asm volatile("s_add_u32 m0, %3, %4\n\ts_cmp_lg_u32 %5, 0\n\ts_cbranch_scc1 1f\n\tbuffer_load_dwordx4 %0, %1, %2 offen sc0 sc1 lds\n1:"
               ::"v"(voff), "s"(srd), "s"(soff), "s"(lds), "n"(LOFF), "s"(skip) : "memory", "scc");
}
__device__ __forceinline__ void dma_touch(i32x4 srd, unsigned voff, unsigned soff, unsigned lds) {
  asm volatile("s_mov_b32 m0, %3\n\ts_nop 0\n\tbuffer_load_dword %0, %1, %2 offen lds" ::"v"(voff), "s"(srd), "s"(soff), "s"(lds) : "memory");
}
__device__ __forceinline__ i32x4 make_srd(u64 a, unsigned bytes) {
  i32x4 r; r.x = (int)(unsigned)a; r.y = (int)((unsigned)(a >> 32) & 0xffffu); r.z = (int)bytes; r.w = 0x00020000; return r;
}
__device__ __forceinline__ void store16(i32x4 srd, unsigned voff, unsigned soff, u32x4 v) {
  asm volatile("buffer_store_dwordx4 %0, %1, %2, %3 offen sc0 sc1\n\ts_nop 1" ::"v"(v), "v"(voff), "s"(srd), "s"(soff) : "memory");
}
__device__ __forceinline__ bool wait_counters(u64 myws, unsigned off, unsigned target, unsigned call, int rank, int l) {
  bool bad = false;
  if (l < NR && l != rank) {
    gu32* f = (gu32*)(myws + off + l * 128);
    gu32* tmo = (gu32*)(myws + WS_TMO);
    const u64 t0 = __builtin_amdgcn_s_memrealtime();
    while (__hip_atomic_load(f, __ATOMIC_RELAXED, __HIP_MEMORY_SCOPE_SYSTEM) < target) {
      if (__hip_atomic_load(tmo, __ATOMIC_RELAXED, __HIP_MEMORY_SCOPE_AGENT) == call || __builtin_amdgcn_s_memrealtime() - t0 > DEADLINE) {
        __hip_atomic_store(tmo, call, __ATOMIC_RELAXED, __HIP_MEMORY_SCOPE_AGENT);
        bad = true;
        break;
      }
      __builtin_amdgcn_s_sleep(1);
    }
  }
  return __builtin_amdgcn_ballot_w64(bad) != 0;
}

struct Lane {
  unsigned xo, wo[4];
  unsigned mw, mx;
  int skip;
  LDSP unsigned char *rd[2][2], *xr[2][2], *pw[2], *pr;
};
template <int I> struct IC { static constexpr int v = I; };

template <int N, int CW>
__device__ __forceinline__ void tile_job(const Lane& L, i32x4 wsrd, i32x4 xsrd, unsigned gs, f32x4 (&sums)[16], u64 asx, u64 asw, u64 acall, float& scale, unsigned& call
) {
  constexpr int NW = 2 * CW, NX = (N + 1) / 2, D = NW + NX, DM = NW + N / 2, M = CW * N;
  f32x4 acc[N][CW];
  i32x4 wf[2][CW][2], xf[2][N][2];
  auto dma = [&](auto slot, auto ic, unsigned st) {
    constexpr int i = decltype(ic)::v, so = decltype(slot)::v * STG;
    if constexpr (i < NW) dma_w<so, i * 1024>(wsrd, L.wo[i], st * (128 * ND), L.mw);
    else if constexpr ((N & 1) && i == D - 1) dma_xs<so + (i - NW) * 4096>(xsrd, L.xo, 2 * gs * (i - NW) + st * 128, L.mx, L.skip);
    else if constexpr (i < D) dma_x<so + (i - NW) * 4096>(xsrd, L.xo, 2 * gs * (i - NW) + st * 128, L.mx);
  };
  auto dman = [&](auto slot, int i, unsigned st) {
    switch (i) {
      case 0: dma(slot, IC<0>{}, st); break; case 1: dma(slot, IC<1>{}, st); break; case 2: dma(slot, IC<2>{}, st); break; case 3: dma(slot, IC<3>{}, st); break;
      case 4: dma(slot, IC<4>{}, st); break; case 5: dma(slot, IC<5>{}, st); break; case 6: dma(slot, IC<6>{}, st); break; default: dma(slot, IC<7>{}, st);
    }
  };
  auto rdw = [&](int slot, int c, int h) {
    LDSP unsigned char* p = L.rd[slot >> 1][c] + (slot & 1) * STG + h * 4096 * CW;
    const i32x2 t0 = __builtin_amdgcn_ds_read_tr8_b64_v2i32((LDSP i32x2*)p);
    const i32x2 t1 = __builtin_amdgcn_ds_read_tr8_b64_v2i32((LDSP i32x2*)(p + 512 * CW));
    wf[slot & 1][c][h] = i32x4{t0.x, t0.y, t1.x, t1.y};
  };
  auto rdx = [&](int slot, int g) {
#pragma unroll
    for (int u = 0; u < 2; ++u) xf[slot & 1][g][u] = *(LDSP i32x4*)(L.xr[slot >> 1][u] + (slot & 1) * STG + g * 2048);
  };
  auto mm = [&](int slot, int g, int c) {
    f32x4 d = __builtin_amdgcn_mfma_scale_f32_16x16x128_f8f6f4(__builtin_shufflevector(wf[slot & 1][c][0], wf[slot & 1][c][1], 0, 1, 2, 3, 4, 5, 6, 7),
                                                               __builtin_shufflevector(xf[slot & 1][g][0], xf[slot & 1][g][1], 0, 1, 2, 3, 4, 5, 6, 7), acc[g][c], 1, 1, 0, 0, 0, 0);
    asm volatile("" : "+a"(d));
    acc[g][c] = d;
  };
  auto pw = [&](int g, int c) { *(LDSP f32x4*)(L.pw[c] + g * 4096 * CW) = acc[g][c]; };
  auto step = [&](auto slot, unsigned st, int left) {
    constexpr int sl = decltype(slot)::v, nx = (sl + 1) & 3;
    if (left >= 2) WAITV(2 * DM); else if (left == 1) WAITV(0);
    if (left >= 1) { __builtin_amdgcn_s_waitcnt(0xC07F); BAR(); }
    SB();
#pragma unroll
    for (int m = 0; m < M; ++m) {
      mm(sl, m / CW, m % CW);
      if (left >= 0) {
#pragma unroll
        for (int i = 0; i < N + CW; ++i) {
          if (m == (M >= 8 && CW == 2 ? i : i / 2)) { if (i < CW) { rdw(nx, i, 0); rdw(nx, i, 1); } else rdx(nx, i - CW); }
        }
      } else if (m >= 2) {
        pw((m - 2) / CW, (m - 2) % CW);
      }
      if (left == 3) {
#pragma unroll
        for (int i = (m * D + M - 1) / M; i < ((m + 1) * D + M - 1) / M; ++i) dman(slot, i, st + 4);
      }
      SB();
    }
  };

#pragma unroll
  for (int i = 0; i < D; ++i) dman(IC<0>{}, i, 0);
#pragma unroll
  for (int i = 0; i < D; ++i) dman(IC<1>{}, i, 1);
#pragma unroll
  for (int i = 2; i < D; ++i) dman(IC<2>{}, i, 2);
  {
    float sx, sw; unsigned d;
    asm volatile("s_load_dword %0, %3, 0x0\n\ts_load_dword %1, %4, 0x0\n\ts_load_dword %2, %5, 0x0\n\t"
                 "s_add_u32 m0, %9, %11\n\ts_nop 0\n\t"
                 "buffer_load_dwordx4 %6, %8, %10 offen lds\n\tbuffer_load_dwordx4 %7, %8, %10 offen offset:1024 lds\n\t"
                 "s_waitcnt lgkmcnt(0)"
                 : "=&s"(sx), "=&s"(sw), "=&s"(d)
                 : "s"(asx), "s"(asw), "s"(acall), "v"(L.wo[0]), "v"(L.wo[1]), "s"(wsrd), "s"(L.mw), "s"(256 * ND), "n"(2 * STG)
                 : "memory", "scc");
    call = d + 1u; scale = sx * sw;
  }
#pragma unroll
  for (int g = 0; g < N; ++g)
#pragma unroll
    for (int c = 0; c < CW; ++c) { f32x4 z = {0.f, 0.f, 0.f, 0.f}; asm volatile("" : "+a"(z)); acc[g][c] = z; }
  WAITV(2 * DM); BAR();
#pragma unroll
  for (int i = 0; i < NW; ++i) rdw(0, i >> 1, i & 1);
#pragma unroll
  for (int g = 0; g < N; ++g) rdx(0, g);
#pragma unroll
  for (int i = 0; i < D; ++i) dman(IC<3>{}, i, 3);
  SB();
#pragma clang loop unroll(disable)
  for (unsigned st = 0; st < 28; st += 4) {
    asm volatile("" : "+s"(st));
    step(IC<0>{}, st, 3); step(IC<1>{}, st + 1, 3); step(IC<2>{}, st + 2, 3); step(IC<3>{}, st + 3, 3);
  }
  step(IC<0>{}, 28, 2); step(IC<1>{}, 29, 1); step(IC<2>{}, 30, 0); step(IC<3>{}, 31, -1);
  pw((M - 2) / CW, (M - 2) % CW); pw(N - 1, CW - 1);
  WAITL(0); BAR();
#pragma unroll
  for (int t = 0; t < M; ++t) sums[t] = *(LDSP f32x4*)(L.pr + t * 4096);
}

__global__ __launch_bounds__(256, 1) void gemm_a2a(Args args) {
  extern __shared__ __attribute__((aligned(16))) unsigned char smem[];
  const int b = blockIdx.x, xcd = b & 7, j = b >> 3;
  const bool local = xcd == 7;
  const CONSTP unsigned char* kp = (const CONSTP unsigned char*)__builtin_amdgcn_kernarg_segment_ptr();
  i32x16 hv = *(const CONSTP i32x16*)kp;
  i32x4 pv = *(const CONSTP i32x4*)(kp + 64 + xcd * 16);
  i32x2 ev = *(const CONSTP i32x2*)(kp + 192);
  unsigned job = *(const CONSTP unsigned*)(kp + 336 + (unsigned)((local ? 128 : 0) + j * 4));
  const int w = __builtin_amdgcn_readfirstlane(threadIdx.x >> 6), l = threadIdx.x & 63;
  int woW = (w * 32 + (l >> 3)) * ND + (((l & 7) ^ ((l >> 4) & 3)) << 4), woN = (w * 32 + (l >> 2)) * ND + (((l & 3) ^ ((l >> 4) & 1)) << 4);
  int xoL = ((w & 1) * 8 + (l >> 3)) * KD + (((l & 7) ^ ((l >> 4) & 3) ^ ((w & 1) << 2)) << 4);
  const int kr = 16 * (l >> 4) + ((l & 15) >> 1);
  int rdW = ((kr * 128 + (((l >> 2) & 7) << 4)) ^ (w << 5)) + (l & 1) * 8, rdN = ((kr * 64 + (((l >> 3) & 3) << 4)) ^ (w << 4)) + (l & 1) * 8;
  int xr = 16384 + (l & 15) * 128 + (((l >> 4) ^ ((l >> 1) & 7)) << 4);
  int paL = (l & 15) * 256 + (((l >> 4) ^ (l & 7)) << 4);
  int prL = (w * 4 + (l >> 4)) * 256 + (((l & 15) ^ ((w & 1) * 4 + (l >> 4))) << 4);
  int ooL = ((w * 4 + (l >> 4)) * NS + (l & 15) * 4) * 4;
  asm volatile("" : "+v"(woW), "+v"(woN), "+v"(xoL), "+v"(rdW), "+v"(rdN), "+v"(xr), "+v"(paL), "+v"(prL), "+v"(ooL));
  asm volatile("" : "+s"(hv), "+s"(pv), "+s"(ev), "+s"(job));
  u64 tstart = 0;
  if (j >= 16 || (job >> 16)) tstart = __builtin_amdgcn_s_memrealtime() - 35;
  auto mk64 = [](int lo, int hi) { return (unsigned)lo | ((u64)(unsigned)hi << 32); };
  const u64 ax = mk64(hv[0], hv[1]), aw = mk64(hv[2], hv[3]), myws = mk64(hv[4], hv[5]), asx = mk64(hv[6], hv[7]), asw = mk64(hv[8], hv[9]);
  const int rank = hv[10], pf = hv[11] & 15, dpf = (hv[11] >> 16) & 255, nraw = hv[13], npk = hv[14];
  const u64 dout = mk64(pv.x, pv.y), dws = mk64(pv.z, pv.w), myout = mk64(ev.x, ev.y);
  const u64 mycall = myws + WS_CALL + b * 128;
  const int dest = (rank + 1 + xcd) & 7;
  const int n = job & 15, g0 = (job >> 4) & 63, pnl = (job >> 10) & 3, tgo = (job >> 16) * 10;
  const bool pk = (job >> 12) & 1, foreign = local && n == 7, wide = !((job >> 14) & 1);
  const int nv = wide ? 2 * n : n;
  const bool late = tgo != 0, tail = (job >> 13) & 1;
  const int c0 = pnl * 128 + ((job >> 15) & 1) * 64, n0 = dest * NS + c0, r0 = g0 * 16;
  const unsigned lds0 = (unsigned)(size_t)(LDSP unsigned char*)smem, reg = lds0 + 4 * STG + w * 256;
  const i32x4 wsrd = make_srd(aw, KD * ND);
  auto until = [&](int ticks) { while ((int)(unsigned)(__builtin_amdgcn_s_memrealtime() - tstart) < ticks) __builtin_amdgcn_s_sleep(1); };
  unsigned call;
  float scale;
  auto get_call = [&]() {
    unsigned d;
    asm volatile("s_load_dword %0, %1, 0x0\n\ts_waitcnt lgkmcnt(0)" : "=&s"(d) : "s"(mycall) : "memory");
    call = d + 1u;
  };
  bool lost = false;

  if (j >= 16) {
    const i32x4 xall = make_srd(ax, MS * KD);
    auto pf_w = [&](int p, int c) {
      dma_touch(wsrd, (unsigned)(c * 64 + l) * ND + dest * NS + p * 128, 0, reg);
    };
    auto pf_x = [&](int kb, int c) {
      dma_touch(xall, (unsigned)(c * 16 + (l & 15)) * KD + (l >> 4) * 128 + kb * 512, 0, reg);
    };
    const int hw = (j - 16) * 4 + w;
    if (j == 27) {
      u64 pc;
      asm volatile("s_getpc_b64 %0" : "=s"(pc));
      dma_touch(make_srd(pc & ~127ull, CODE_BYTES), (w * 64 + l) * 128, 0, reg);
    }
    if (local) {
      if (pf & 4) {
        until(dpf * 10);
#pragma unroll
        for (int kb = 0; kb < 8; ++kb) { if (hw < 32) pf_w(hw & 3, kb * 8 + (hw >> 2)); else pf_x(kb, hw - 32); }
      }
      get_call();
      lost = wait_counters(myws, WS_XCNT, call * 4u, call, rank, l);
    } else if (j >= 28) {
      u32x4 v[16];
      const unsigned poff = (hw - 48) * KD + l * 16;
      auto ld = [&](int i) {
        asm volatile("buffer_load_dwordx4 %0, %1, %2, %3 offen offset:%4" : "=&v"(v[i]) : "v"(poff), "s"(xall), "s"((unsigned)(448 + (i >> 2) * 16) * KD), "n"((i & 3) * 1024) : "memory");
      };
#pragma unroll
      for (int i = 0; i < 6; ++i) ld(i);
      get_call();
      const i32x4 psrd = make_srd(dws + WS_XS + (call & 1) * XS_PAR + xcd * XS_SRC, 64 * KD);
#pragma unroll
      for (int i = 0; i < 16; ++i) {
        const int younger = i < 6 ? 5 + i : i <= 10 ? 11 : 21 - i;
        switch (younger) {
          case 5: WAITV(5); break; case 6: WAITV(6); break; case 7: WAITV(7); break; case 8: WAITV(8); break;
          case 9: WAITV(9); break; case 10: WAITV(10); break; default: WAITV(11); break;
        }
        asm volatile("" : "+v"(v[i]));
        if (i + 6 < 16) ld(i + 6);
        asm volatile("buffer_store_dwordx4 %0, %1, %2, %3 offen offset:%4 sc0 sc1\n\ts_nop 1" ::"v"(v[i]), "v"(poff), "s"(psrd), "s"((i >> 2) * 16 * KD), "n"((i & 3) * 1024) : "memory");
        if (i == 0) STAMP(8);
      }
      WAITV(0);
      BAR();
      STAMP(9);
      if (threadIdx.x == 0) {
        __hip_atomic_fetch_add((gu32*)(dws + WS_XCNT + rank * 128), 1u, __ATOMIC_RELAXED, __HIP_MEMORY_SCOPE_SYSTEM);
      }
    } else {
      if ((pf & 1) && hw < 44) {
#pragma unroll
        for (int kb = 0; kb < 8; ++kb) { if (hw < 16) pf_w(hw & 1, kb * 8 + (hw >> 1)); else pf_x(kb, hw - 16); }
      }
      if (pf & 2) {
        until(tgo - 80);
        pf_w(2 + (hw & 1), hw >> 1);
      }
    }
    if (!local && j >= 28 && (pf & 2)) {
#pragma unroll
      for (int q = 3; q < 8; ++q) pf_w(2 + (hw & 1), q * 8 + ((hw - 48) >> 1));
      WAITV(0);
    }
  }
  auto row_exp = [&](f32x4 v) {
    unsigned m = __builtin_bit_cast(unsigned, fmaxf(fmaxf(fabsf(v.x), fabsf(v.y)), fmaxf(fabsf(v.z), fabsf(v.w))));
    m = max(m, (unsigned)__builtin_amdgcn_update_dpp(0, (int)m, 0x128, 0xf, 0xf, true));
    m = max(m, (unsigned)__builtin_amdgcn_update_dpp(0, (int)m, 0x124, 0xf, 0xf, true));
    m = max(m, (unsigned)__builtin_amdgcn_update_dpp(0, (int)m, 0x122, 0xf, 0xf, true));
    m = max(m, (unsigned)__builtin_amdgcn_update_dpp(0, (int)m, 0x121, 0xf, 0xf, true));
    return (int)((m >> 23) & 255);
  };
  auto enc = [&](f32x4 v, int E) {
    const f32x4 t = {__builtin_amdgcn_ldexpf(v.x, 1 - E), __builtin_amdgcn_ldexpf(v.y, 1 - E), __builtin_amdgcn_ldexpf(v.z, 1 - E), __builtin_amdgcn_ldexpf(v.w, 1 - E)};
    return __builtin_bit_cast(u32x4, t) & 0x80ffffffu;
  };
  auto dec = [&](u32x4 y, int E) {
    const f32x4 t = __builtin_bit_cast(f32x4, y);
    return f32x4{__builtin_amdgcn_ldexpf(t.x, E - 1), __builtin_amdgcn_ldexpf(t.y, E - 1), __builtin_amdgcn_ldexpf(t.z, E - 1), __builtin_amdgcn_ldexpf(t.w, E - 1)};
  };

  const unsigned poff = w * 4096 + l * 16;
  if (n) {
  if (tgo) until(tgo);
  STAMP(1);

  Lane L;
  const unsigned gs = foreign ? XS_SRC : 16 * KD;
  L.xo = xoL + (w >> 1) * gs;
  L.wo[0] = wide ? woW : woN;
  L.wo[1] = wide ? woW + 8 * ND - 1024 : (woN ^ 32) + 16 * ND - 1024;
  L.wo[2] = (woW ^ 64) + 16 * ND - 2048; L.wo[3] = (woW ^ 64) + 24 * ND - 3072;
  L.mw = lds0 + (w << (wide ? 12 : 11)); L.mx = lds0 + 16384 + w * 1024;
  L.skip = w >> 1;
  {
    LDSP unsigned char* const base = (LDSP unsigned char*)smem;
    const int rd = wide ? rdW : rdN, pa = paL ^ (wide ? (w >> 1) * 4096 + (w & 1) * 128 : w * 64);
#pragma unroll
    for (int hf = 0; hf < 2; ++hf) {
      int a0 = rd + hf * 2 * STG, a1 = a0 ^ 16, a2 = xr + hf * 2 * STG, a3 = a2 ^ 64;
      asm("" : "+v"(a0), "+v"(a1), "+v"(a2), "+v"(a3));
      L.rd[hf][0] = base + a0; L.rd[hf][1] = base + a1; L.xr[hf][0] = base + a2; L.xr[hf][1] = base + a3;
    }
    L.pw[0] = base + pa; L.pw[1] = base + (pa ^ 64);
    L.pr = base + prL;
  }
  const unsigned ooff = ooL + c0 * 4;
  const i32x4 osrd = make_srd(dout, NR * MS * NS * 4);

#define CYARG
  f32x4 res[16];
  {
    unsigned par = 0;
    if (foreign) par = call & 1;
    const i32x4 wsrd = make_srd(aw + n0, KD * ND - n0);
    const u64 xown = ax + (unsigned)r0 * KD;
    const i32x4 xs = make_srd(xown + (foreign ? myws + (WS_XS + par * XS_PAR + (g0 - 28) * 16 * KD) - xown : 0ull), foreign ? 7 * XS_SRC : n * 16 * KD);
    const unsigned row0 = (unsigned)(rank * MS + r0) * (NS * 4);
    auto quick = [&](int cnt, int cw) {
      STAMP(11); STAMP(3);
#pragma unroll
      for (int t = 0; t < cnt; ++t) store16(osrd, ooff, row0 + (t / cw) * 16 * NS * 4 + (t % cw) * 256, __builtin_bit_cast(u32x4, res[t] * scale));
    };
#define JOB(N, CW) tile_job<N, CW>(L, wsrd, xs, gs, res, asx, asw, mycall, scale, call CYARG)
    if (!wide) {
      if (n == 4) { JOB(4, 1); quick(4, 1); }
      else JOB(5, 1);
    } else {
      if (n == 2) { JOB(2, 2); quick(4, 2); }
      else if (n == 3) JOB(3, 2);
      else if (n == 5) JOB(5, 2);
      else if (n == 6) JOB(6, 2);
      else if (n == 7) JOB(7, 2);
      else JOB(8, 2);
    }
  }
  if (nv > 4) {
  if (lost) scale = __builtin_nanf("");
  STAMP(11);
  auto voff = [&](int t) { return wide ? (t >> 1) * 16 * NS * 4 + (t & 1) * 256 : t * 16 * NS * 4; };
  auto orow = [&](int t) { return foreign ? (unsigned)(((rank + 7 - (t >> 1)) & 7) * MS + r0) * (NS * 4) + (t & 1) * 256 : (unsigned)(rank * MS + r0) * (NS * 4) + voff(t); };
  STAMP(3);
  const int units = pk ? nv / 5 : 0;
  auto rest = [&]() {
#pragma unroll
    for (int t = 0; t < 16; ++t) {
      if (t < nv && t >= 5 * units) {
        store16(osrd, ooff, orow(t), __builtin_bit_cast(u32x4, res[t] * scale));
        LSTAMP(t);
      }
    }
  };
  if (pk && !late && nv == 6) store16(osrd, ooff, orow(5), __builtin_bit_cast(u32x4, res[5] * scale));
  if (pk) {
    const i32x4 ps = make_srd(dws + (WS_PK + (call & 1) * PK_PAR + rank * PK_SRC + j * PK_WG), PK_WG);
#pragma unroll
    for (int u = 0; u < 2; ++u) {
      if (u < units) {
        int E[5];
        E[0] = row_exp(res[5 * u]);
        const u32x4 y0 = enc(res[5 * u], E[0]);
        u32x4 hid = (y0 & 0x00ffffffu) | ((y0 >> 7) & 0x01000000u);
#pragma unroll
        for (int g = 1; g < 4; ++g) {
          E[g] = row_exp(res[5 * u + g]);
          const u32x4 v = enc(res[5 * u + g], E[g]) | (((hid >> (7 * (g - 1))) & 127u) << 24);
          if (late) WAITV(WIN - 1); else WAITV(WIN1 - 1);
          store16(ps, poff, u * PK_RUN + (g - 1) * 1024, v);
          LSTAMP(5 * u + g);
        }
        E[4] = row_exp(res[5 * u + 4]);
        unsigned side = 0;
#pragma unroll
        for (int g = 0; g < 5; ++g) {
          const u32x4 d = __builtin_bit_cast(u32x4, dec(enc(res[5 * u + g], E[g]), E[g])) ^ __builtin_bit_cast(u32x4, res[5 * u + g]);
          side |= ((d.x | d.y | d.z | d.w) ? 1u : 0u) << g;
          side |= ((l >> 1) & 7) == g ? ((E[g] >> ((l & 1) * 4)) & 15) << 8 : 0;
        }
        hid.x |= (side & 7) << 25; hid.y |= ((side >> 3) & 7) << 25; hid.z |= ((side >> 6) & 7) << 25; hid.w |= (side >> 9) << 25;
        if (late) WAITV(WIN - 1); else WAITV(WIN1 - 1);
        store16(ps, poff, u * PK_RUN + 3 * 1024, enc(res[5 * u + 4], E[4]) | ((hid >> 21) << 24));
        LSTAMP(5 * u + 4);
        if (__builtin_amdgcn_ballot_w64(side & 31)) {
#pragma unroll
          for (int g = 0; g < 5; ++g) {
            const bool e = (side >> g) & 1;
            if (__builtin_amdgcn_ballot_w64(e)) store16(osrd, e ? ooff : OOB, orow(5 * u + g), __builtin_bit_cast(u32x4, res[5 * u + g] * scale));
          }
        }
      }
    }
    WAITV(0);
    if (l == 0) __hip_atomic_store((gu32*)(dws + WS_PKF + ((rank * 32 + j) * 4 + w) * 128), call, __ATOMIC_RELAXED, __HIP_MEMORY_SCOPE_SYSTEM);
    STAMP(2);
  }
  if (tail) rest();
  }
  STAMP(4);
  if (tail) WAITV(0);
  BAR();
  STAMP(5);
  if (threadIdx.x == 0) {
    if (tail && !local) __hip_atomic_fetch_add((gu32*)(dws + WS_OCNT + rank * 128), 1u, __ATOMIC_RELAXED, __HIP_MEMORY_SCOPE_SYSTEM);
    __hip_atomic_store((gu32*)mycall, call, __ATOMIC_RELAXED, __HIP_MEMORY_SCOPE_AGENT);
  }
  if (local && j == 0 && w == 2) {
    if (wait_counters(myws, WS_OCNT, call * nraw, call, rank, l) && l == 0)
      ((gf32*)dout)[(size_t)(rank * MS + r0) * NS + c0] = __builtin_nanf("");
  }
  } else {
    float sx, sw; unsigned d;
    asm volatile("s_load_dword %0, %3, 0x0\n\ts_load_dword %1, %4, 0x0\n\ts_load_dword %2, %5, 0x0\n\ts_waitcnt lgkmcnt(0)"
                 : "=&s"(sx), "=&s"(sw), "=&s"(d) : "s"(asx), "s"(asw), "s"(mycall) : "memory");
    call = d + 1u; scale = sx * sw;
    WAITV(0);
    BAR();
    if (threadIdx.x == 0) __hip_atomic_store((gu32*)mycall, call, __ATOMIC_RELAXED, __HIP_MEMORY_SCOPE_AGENT);
  }
  if (!local) {
    const int src = (rank + 7 - xcd) & 7;
    gu32* tmo = (gu32*)(myws + WS_TMO);
#pragma clang loop unroll(disable)
    for (unsigned list = *(const CONSTP unsigned*)(kp + 208 + j * 4); list & 0x80; list >>= 8) {
      const int js = (list >> 1) & 31, u = list & 1;
      const unsigned jw = *(const CONSTP unsigned*)(kp + 336 + js * 4);
      const bool uw = !((jw >> 14) & 1);
      const i32x4 isrd = make_srd(myws + (WS_PK + (call & 1) * PK_PAR + src * PK_SRC + js * PK_WG + u * PK_RUN), PK_RUN);
      const i32x4 msrd = make_srd(myout + ((unsigned)(src * MS + ((jw >> 4) & 63) * 16) * NS + ((jw >> 10) & 3) * 128 + ((jw >> 15) & 1) * 64) * 4, 128 * NS * 4);
      unsigned vo[5];
#pragma unroll
      for (int g = 0; g < 5; ++g) { const int t = 5 * u + g; vo[g] = uw ? (t >> 1) * 16 * NS * 4 + (t & 1) * 256 : t * 16 * NS * 4; }
      gu32* f = (gu32*)(myws + WS_PKF + ((src * 32 + js) * 4 + w) * 128);
      const u64 t0 = __builtin_amdgcn_s_memrealtime();
      bool bad = false;
      for (unsigned spin = 1; __hip_atomic_load(f, __ATOMIC_RELAXED, __HIP_MEMORY_SCOPE_SYSTEM) < call; ++spin) {
        if ((spin & 15) == 0 && (__hip_atomic_load(tmo, __ATOMIC_RELAXED, __HIP_MEMORY_SCOPE_AGENT) == call || __builtin_amdgcn_s_memrealtime() - t0 > DEADLINE)) {
          __hip_atomic_store(tmo, call, __ATOMIC_RELAXED, __HIP_MEMORY_SCOPE_AGENT);
          bad = true;
          break;
        }
        __builtin_amdgcn_s_sleep(PSLEEP);
      }
      STAMP(12);
      const float sc = bad ? __builtin_nanf("") : scale;
      u32x4 c[4];
      asm volatile("buffer_load_dwordx4 %0, %4, %5, 0 offen sc0 sc1\n\tbuffer_load_dwordx4 %1, %4, %5, 0 offen offset:1024 sc0 sc1\n\t"
                   "buffer_load_dwordx4 %2, %4, %5, 0 offen offset:2048 sc0 sc1\n\tbuffer_load_dwordx4 %3, %4, %5, 0 offen offset:3072 sc0 sc1\n\ts_waitcnt vmcnt(0)"
                   : "=&v"(c[0]), "=&v"(c[1]), "=&v"(c[2]), "=&v"(c[3]) : "v"(poff), "s"(isrd) : "memory");
      const int side = bad ? 0 : (int)(((c[3].x >> 28) & 7) | (((c[3].y >> 28) & 7) << 3) | (((c[3].z >> 28) & 7) << 6) | (((c[3].w >> 28) & 7) << 9));
      const int nib = side >> 8;
      auto put = [&](int g, u32x4 y, int E) { store16(msrd, ((side >> g) & 1) ? OOB : ooL, vo[g], __builtin_bit_cast(u32x4, dec(y, E) * sc)); };
#define PUT(g, y) put(g, y, __builtin_amdgcn_update_dpp(0, nib, 0x150 + 2 * g, 0xf, 0xf, false) | (__builtin_amdgcn_update_dpp(0, nib, 0x151 + 2 * g, 0xf, 0xf, false) << 4))
      PUT(1, c[0] & 0x80ffffffu); PUT(2, c[1] & 0x80ffffffu); PUT(3, c[2] & 0x80ffffffu); PUT(4, c[3] & 0x80ffffffu);
      u32x4 h = ((c[3] >> 3) & 0x00e00000u) | ((c[3] << 4) & 0x80000000u);
#pragma unroll
      for (int k = 0; k < 3; ++k) h |= ((c[k] >> 24) & 127u) << (7 * k);
      PUT(0, h);
    }
  }
}

extern "C" void kernel_launch(int rank, int nranks, ncclComm_t comm, hipStream_t stream, void* const* d_in, const int* in_sizes, int n_in,
                              void* d_out, int out_size, const int* in_dtypes, int out_dtype, void* const* d_out_all, void* const* d_ws_all, size_t ws_bytes) {
  static bool init = false;
  if (!init) { (void)hipFuncSetAttribute((const void*)gemm_a2a, hipFuncAttributeMaxDynamicSharedMemorySize, LDS_BYTES); init = true; }
  Args a;
  for (int i = 0; i < NR; ++i) { int p = (rank + 1 + i) & 7; a.peer[i].out = d_out_all[p]; a.peer[i].ws = d_ws_all[p]; }
  a.x = d_in[0]; a.w = d_in[1]; a.sx = d_in[2]; a.sw = d_in[3];
  a.rank = rank; a.myws = d_ws_all[rank]; a.myout = d_out; a.pad0 = a.pad1 = a.pad2 = 0;
  auto env = [](const char* n, int d) { return d; };
  a.cfg = env("PF", 7) | (env("WIN", 6) << 4) | (env("TWIN", 6) << 8) | (env("DPF", 55) << 16);
  const char* plan[4] = {"22335553", "2335555", "66655", "66655"};
  const int d2 = env("D2", 40), dl = env("DL", 65), pkon = env("PK", 1);
  a.nraw = a.npk = 0;
  for (int j = 0; j < 64; ++j) a.job[j] = 0;
  int j = 0;
  auto add = [&](int n, int g0, int p, int narrow, int half, int held, int t) {
    const int nv = narrow ? n : 2 * n, pk = pkon && !held && nv >= 5;
    const int tail = !pk || (nv % 5 && !(t == 0 && nv == 6));
    a.nraw += tail;
    a.job[j++] = n | (g0 << 4) | (p << 10) | (pk << 12) | (tail << 13) | (narrow << 14) | (half << 15) | (t << 16);
  };
  for (int p = 0; p < 4; ++p) {
    int g = 0;
    if (p == 2) j = 16;
    const int t = p < 2 ? 0 : d2;
    for (const char* c = plan[p]; *c; ++c) {
      if (*c == 'n') { ++c; add(*c - '0', g, p, 1, 0, 0, t); add(*c - '0', g, p, 1, 1, 0, t); }
      else if (*c == 'h') { ++c; add(*c - '0', g, p, 0, 0, 1, t); }
      else add(*c - '0', g, p, 0, 0, 0, t);
      g += *c - '0';
    }
    if (g != 28 || j > (p < 2 ? 16 : 28)) { fprintf(stderr, "bad plan\n"); return; }
  }
  for (; j < 28; ++j) a.job[j] = d2 << 16;
  for (j = 0; j < 32; ++j) a.job[32 + j] = j < 16 ? (8 | ((8 * (j >> 2)) << 4) | ((j & 3) << 10) | 8192 | (dl << 16)) : (7 | ((28 + ((j - 16) >> 2)) << 4) | ((j & 3) << 10) | 8192);
  a.hold = 0;
  int cnt[32] = {0};
  for (int q = 0; q < 32; ++q) a.unp[q] = 0;
  for (int wave = 0; wave < 2; ++wave) {
    int freew[32], nf = 0, k = 0;
    for (int q = 31; q >= 0; --q) { const int n = a.job[q] & 15, nv = (a.job[q] >> 14 & 1) ? n : 2 * n; if (n == 0 || (q < 16 && (wave || nv <= 6))) freew[nf++] = q; }
    for (int big = 0; big < 2; ++big)
      for (int q = 16 * wave; q < 16 * wave + 16; ++q) {
        if (!(a.job[q] >> 12 & 1)) continue;
        const int n = a.job[q] & 15, nv = (a.job[q] >> 14 & 1) ? n : 2 * n;
        if ((nv >= 10) != big) continue;
        for (int u = 0; u < nv / 5; ++u) {
          const int t = freew[k++ % nf];
          if (cnt[t] >= 4) { fprintf(stderr, "bad unpack list\n"); return; }
          a.unp[t] |= (unsigned)(0x80 | (q << 1) | u) << (8 * cnt[t]++);
        }
      }
  }
  gemm_a2a<<<256, 256, LDS_BYTES, stream>>>(a);
}
```

```cpp
#include <hip/hip_runtime.h>
#include <rccl/rccl.h>
#include <cstdio>
#include <cstdlib>

typedef unsigned u32x4 __attribute__((ext_vector_type(4)));
typedef int i32x2 __attribute__((ext_vector_type(2)));
typedef int i32x4 __attribute__((ext_vector_type(4)));
typedef int i32x8 __attribute__((ext_vector_type(8)));
typedef int i32x16 __attribute__((ext_vector_type(16)));
typedef float f32x4 __attribute__((ext_vector_type(4)));
typedef unsigned long long u64;
#define LDSP __attribute__((address_space(3)))
#define CONSTP __attribute__((address_space(4)))
typedef __attribute__((address_space(1))) unsigned gu32;
typedef __attribute__((address_space(1))) float gf32;
typedef __attribute__((address_space(1))) u64 gu64;

constexpr int NR = 8, MS = 512, KD = 4096, ND = 4096, NS = 512;
constexpr int STG = 32768;
constexpr int LDS_BYTES = 4 * STG + 1024;
constexpr u64 DEADLINE = 20000000ull;
constexpr unsigned WS_TMO = 128, WS_OCNT = 4096, WS_XCNT = 8192, WS_TAIL = 12288, WS_CALL = 65536, WS_PKF = 131072, WS_XS = 524288;
constexpr unsigned XS_SRC = 64 * KD, XS_PAR = NR * XS_SRC;
constexpr unsigned WS_PK = 8u << 20, PK_RUN = 16384, PK_WG = 2 * PK_RUN, PK_SRC = 32 * PK_WG, PK_PAR = NR * PK_SRC;
constexpr unsigned OOB = 0x40000000u;
#define TLEFT 6
#define WIN 4
#define FT 0
#define CNT 1
#define WINB 4
#define WINC 4
#define WIN1 9
#define PSLEEP 1
#define TWIN 8
#define CODE_BYTES 30720
#define STAMP(i)
#define LSTAMP(i)

struct Args {
  const void* x; const void* w; void* myws; const void* sx; const void* sw;
  int rank, cfg, hold, nraw, npk, pad0;
  struct { void* out; void* ws; } peer[NR];
  void* myout; int pad1, pad2;
  unsigned unp[32];
  unsigned job[64];
  unsigned short pfl[48][8];
};

#define WAITV(n) asm volatile("s_waitcnt vmcnt(%0)" ::"n"(n) : "memory")
#define WAITL(n) asm volatile("s_waitcnt lgkmcnt(%0)" ::"n"(n) : "memory")
#define SB() __builtin_amdgcn_sched_barrier(0)
#define BAR() do { __builtin_amdgcn_s_barrier(); asm volatile("" ::: "memory"); } while (0)

template <int LOFF, int OFF>
__device__ __forceinline__ void dma_w(i32x4 srd, unsigned voff, unsigned soff, unsigned lds) {
  asm volatile("s_add_u32 m0, %4, %5\n\ts_nop 0\n\tbuffer_load_dwordx4 %0, %1, %2 offen offset:%3 lds" ::"v"(voff), "s"(srd), "s"(soff), "n"(OFF), "s"(lds), "n"(LOFF) : "memory", "scc");
}
template <int LOFF>
__device__ __forceinline__ void dma_x(i32x4 srd, unsigned voff, unsigned soff, unsigned lds) {
  asm volatile("s_add_u32 m0, %3, %4\n\ts_nop 0\n\tbuffer_load_dwordx4 %0, %1, %2 offen sc0 sc1 lds" ::"v"(voff), "s"(srd), "s"(soff), "s"(lds), "n"(LOFF) : "memory", "scc");
}
template <int LOFF>
__device__ __forceinline__ void dma_xs(i32x4 srd, unsigned voff, unsigned soff, unsigned lds, int skip) {
  asm volatile("s_add_u32 m0, %3, %4\n\ts_cmp_lg_u32 %5, 0\n\ts_cbranch_scc1 1f\n\tbuffer_load_dwordx4 %0, %1, %2 offen sc0 sc1 lds\n1:"
               ::"v"(voff), "s"(srd), "s"(soff), "s"(lds), "n"(LOFF), "s"(skip) : "memory", "scc");
}
__device__ __forceinline__ void dma_touch(i32x4 srd, unsigned voff, unsigned soff, unsigned lds) {
  asm volatile("s_mov_b32 m0, %3\n\ts_nop 0\n\tbuffer_load_dword %0, %1, %2 offen lds" ::"v"(voff), "s"(srd), "s"(soff), "s"(lds) : "memory");
}
__device__ __forceinline__ i32x4 make_srd(u64 a, unsigned bytes) {
  i32x4 r; r.x = (int)(unsigned)a; r.y = (int)((unsigned)(a >> 32) & 0xffffu); r.z = (int)bytes; r.w = 0x00020000; return r;
}
__device__ __forceinline__ void store16(i32x4 srd, unsigned voff, unsigned soff, u32x4 v) {
  asm volatile("buffer_store_dwordx4 %0, %1, %2, %3 offen sc0 sc1\n\ts_nop 1" ::"v"(v), "v"(voff), "s"(srd), "s"(soff) : "memory");
}
template <int OFF>
__device__ __forceinline__ void gstore16(u64 base, unsigned voff, u32x4 v) {
  asm volatile("global_store_dwordx4 %0, %1, %2 offset:%3 sc0 sc1\n\ts_nop 1" ::"v"(voff), "v"(v), "s"(base), "n"(OFF) : "memory");
}
__device__ __forceinline__ bool wait_counters(u64 myws, unsigned off, unsigned target, unsigned call, int rank, int l) {
  bool bad = false;
  if (l < NR && l != rank) {
    gu32* f = (gu32*)(myws + off + l * 128);
    gu32* tmo = (gu32*)(myws + WS_TMO);
    const u64 t0 = __builtin_amdgcn_s_memrealtime();
    while (__hip_atomic_load(f, __ATOMIC_RELAXED, __HIP_MEMORY_SCOPE_SYSTEM) < target) {
      if (__hip_atomic_load(tmo, __ATOMIC_RELAXED, __HIP_MEMORY_SCOPE_AGENT) == call || __builtin_amdgcn_s_memrealtime() - t0 > DEADLINE) {
        __hip_atomic_store(tmo, call, __ATOMIC_RELAXED, __HIP_MEMORY_SCOPE_AGENT);
        bad = true;
        break;
      }
      __builtin_amdgcn_s_sleep(1);
    }
  }
  return __builtin_amdgcn_ballot_w64(bad) != 0;
}

struct Lane {
  unsigned xo, wo[4];
  unsigned mw, mx;
  int skip;
  LDSP unsigned char *rd[2][2], *xr[2][2], *pw[2], *pr;
};
template <int I> struct IC { static constexpr int v = I; };

template <int N, int CW>
__device__ __forceinline__ void tile_job(const Lane& L, i32x4 wsrd, i32x4 xsrd, unsigned gs, f32x4 (&sums)[16], u64 asx, u64 asw, u64 acall, float& scale, unsigned& call
) {
  constexpr int NW = 2 * CW, NX = (N + 1) / 2, D = NW + NX, DM = NW + N / 2, M = CW * N;
  f32x4 acc[N][CW];
  i32x4 wf[4][CW][2], xf[4][N][2];
  auto dma = [&](auto slot, auto ic, unsigned st) {
    constexpr int i = decltype(ic)::v, so = decltype(slot)::v * STG;
    if constexpr (i < NW) dma_w<so, i * 1024>(wsrd, L.wo[i], st * (128 * ND), L.mw);
    else if constexpr ((N & 1) && i == D - 1) dma_xs<so + (i - NW) * 4096>(xsrd, L.xo, 2 * gs * (i - NW) + st * 128, L.mx, L.skip);
    else if constexpr (i < D) dma_x<so + (i - NW) * 4096>(xsrd, L.xo, 2 * gs * (i - NW) + st * 128, L.mx);
  };
  auto dman = [&](auto slot, int i, unsigned st) {
    switch (i) {
      case 0: dma(slot, IC<0>{}, st); break; case 1: dma(slot, IC<1>{}, st); break; case 2: dma(slot, IC<2>{}, st); break; case 3: dma(slot, IC<3>{}, st); break;
      case 4: dma(slot, IC<4>{}, st); break; case 5: dma(slot, IC<5>{}, st); break; case 6: dma(slot, IC<6>{}, st); break; default: dma(slot, IC<7>{}, st);
    }
  };
  auto rdw = [&](int slot, int c, int h) {
    LDSP unsigned char* p = L.rd[slot >> 1][c] + (slot & 1) * STG + h * 4096 * CW;
    const i32x2 t0 = __builtin_amdgcn_ds_read_tr8_b64_v2i32((LDSP i32x2*)p);
    const i32x2 t1 = __builtin_amdgcn_ds_read_tr8_b64_v2i32((LDSP i32x2*)(p + 512 * CW));
    wf[slot][c][h] = i32x4{t0.x, t0.y, t1.x, t1.y};
  };
  auto rdx = [&](int slot, int g) {
#pragma unroll
    for (int u = 0; u < 2; ++u) xf[slot][g][u] = *(LDSP i32x4*)(L.xr[slot >> 1][u] + (slot & 1) * STG + g * 2048);
  };
  auto mm = [&](int slot, int g, int c) {
    f32x4 d = __builtin_amdgcn_mfma_scale_f32_16x16x128_f8f6f4(__builtin_shufflevector(wf[slot][c][0], wf[slot][c][1], 0, 1, 2, 3, 4, 5, 6, 7),
                                                               __builtin_shufflevector(xf[slot][g][0], xf[slot][g][1], 0, 1, 2, 3, 4, 5, 6, 7), acc[g][c], 1, 1, 0, 0, 0, 0);
    asm volatile("" : "+a"(d));
    acc[g][c] = d;
  };
  auto pw = [&](int g, int c) { *(LDSP f32x4*)(L.pw[c] + g * 4096 * CW) = acc[g][c]; };
  auto step = [&](auto slot, unsigned st, int left) {
    constexpr int sl = decltype(slot)::v, nx = (sl + 1) & 3;
    if (left >= 2) WAITV(2 * DM); else if (left == 1) WAITV(0);
    if (left >= 1) { __builtin_amdgcn_s_waitcnt(0xC07F); BAR(); }
    SB();
#pragma unroll
    for (int m = 0; m < M; ++m) {
      mm(sl, m / CW, m % CW);
      if (left >= 0) {
#pragma unroll
        for (int i = 0; i < N + CW; ++i) {
          if (m == (M >= 8 && CW == 2 ? i : i / 2)) { if (i < CW) { rdw(nx, i, 0); rdw(nx, i, 1); } else rdx(nx, i - CW); }
        }
      } else if (m >= 2) {
        pw((m - 2) / CW, (m - 2) % CW);
      }
      if (left == 3) {
#pragma unroll
        for (int i = (m * D + M - 1) / M; i < ((m + 1) * D + M - 1) / M; ++i) dman(slot, i, st + 4);
      }
      SB();
    }
  };

#pragma unroll
  for (int i = 0; i < D; ++i) dman(IC<0>{}, i, 0);
#pragma unroll
  for (int i = 0; i < D; ++i) dman(IC<1>{}, i, 1);
#pragma unroll
  for (int i = 0; i < D; ++i) dman(IC<2>{}, i, 2);
#pragma unroll
  for (int g = 0; g < N; ++g)
#pragma unroll
    for (int c = 0; c < CW; ++c) { f32x4 z = {0.f, 0.f, 0.f, 0.f}; asm volatile("" : "+a"(z)); acc[g][c] = z; }
#pragma unroll
  for (int i = 2; i < D; ++i) dman(IC<3>{}, i, 3);
  {
    float sx, sw; unsigned d;
    asm volatile("s_load_dword %0, %3, 0x0\n\ts_load_dword %1, %4, 0x0\n\ts_load_dword %2, %5, 0x0\n\t"
                 "s_add_u32 m0, %9, %11\n\ts_nop 0\n\t"
                 "buffer_load_dwordx4 %6, %8, %10 offen lds\n\tbuffer_load_dwordx4 %7, %8, %10 offen offset:1024 lds\n\t"
                 "s_waitcnt lgkmcnt(0)"
                 : "=&s"(sx), "=&s"(sw), "=&s"(d)
                 : "s"(asx), "s"(asw), "s"(acall), "v"(L.wo[0]), "v"(L.wo[1]), "s"(wsrd), "s"(L.mw), "s"(384 * ND), "n"(3 * STG)
                 : "memory", "scc");
    call = d + 1u; scale = sx * sw;
  }
  WAITV(3 * DM); BAR();
#pragma unroll
  for (int i = 0; i < NW; ++i) rdw(0, i >> 1, i & 1);
#pragma unroll
  for (int g = 0; g < N; ++g) rdx(0, g);
  SB();
#pragma clang loop unroll(disable)
  for (unsigned st = 0; st < 28; st += 4) {
    asm volatile("" : "+s"(st));
    step(IC<0>{}, st, 3); step(IC<1>{}, st + 1, 3); step(IC<2>{}, st + 2, 3); step(IC<3>{}, st + 3, 3);
  }
  if (M <= FT) {
    auto rd = [&](int slot) {
#pragma unroll
      for (int i = 0; i < NW; ++i) rdw(slot, i >> 1, i & 1);
#pragma unroll
      for (int g = 0; g < N; ++g) rdx(slot, g);
    };
    auto mms = [&](int slot) {
#pragma unroll
      for (int m = 0; m < M; ++m) mm(slot, m / CW, m % CW);
    };
    WAITV(DM); BAR();
    rd(1); rd(2); SB(); mms(0); mms(1); SB();
    WAITV(0); BAR();
    rd(3); SB(); mms(2); SB();
    step(IC<3>{}, 31, -1);
  } else {
    step(IC<0>{}, 28, 2); step(IC<1>{}, 29, 1); step(IC<2>{}, 30, 0); step(IC<3>{}, 31, -1);
  }
  pw((M - 2) / CW, (M - 2) % CW); pw(N - 1, CW - 1);
  WAITL(0); BAR();
#pragma unroll
  for (int t = 0; t < M; ++t) sums[t] = *(LDSP f32x4*)(L.pr + t * 4096);
}

__global__ __launch_bounds__(256, 1) void gemm_a2a(Args args) {
  extern __shared__ __attribute__((aligned(16))) unsigned char smem[];
  const int b = blockIdx.x, xcd = b & 7, j = b >> 3;
  const bool local = xcd == 7;
  const CONSTP unsigned char* kp = (const CONSTP unsigned char*)__builtin_amdgcn_kernarg_segment_ptr();
  i32x16 hv = *(const CONSTP i32x16*)kp;
  i32x4 pv = *(const CONSTP i32x4*)(kp + 64 + xcd * 16);
  i32x2 ev = *(const CONSTP i32x2*)(kp + 192);
  unsigned job = *(const CONSTP unsigned*)(kp + 336 + (unsigned)((local ? 128 : 0) + j * 4));
  const int w = __builtin_amdgcn_readfirstlane(threadIdx.x >> 6), l = threadIdx.x & 63;
  int woW = (w * 32 + (l >> 3)) * ND + (((l & 7) ^ ((l >> 4) & 3)) << 4), woN = (w * 32 + (l >> 2)) * ND + (((l & 3) ^ ((l >> 4) & 1)) << 4);
  int xoL = ((w & 1) * 8 + (l >> 3)) * KD + (((l & 7) ^ ((l >> 4) & 3) ^ ((w & 1) << 2)) << 4);
  const int kr = 16 * (l >> 4) + ((l & 15) >> 1);
  int rdW = ((kr * 128 + (((l >> 2) & 7) << 4)) ^ (w << 5)) + (l & 1) * 8, rdN = ((kr * 64 + (((l >> 3) & 3) << 4)) ^ (w << 4)) + (l & 1) * 8;
  int xr = 16384 + (l & 15) * 128 + (((l >> 4) ^ ((l >> 1) & 7)) << 4);
  int paL = (l & 15) * 256 + (((l >> 4) ^ (l & 7)) << 4);
  int prL = (w * 4 + (l >> 4)) * 256 + (((l & 15) ^ ((w & 1) * 4 + (l >> 4))) << 4);
  int ooL = ((w * 4 + (l >> 4)) * NS + (l & 15) * 4) * 4;
  asm volatile("" : "+v"(woW), "+v"(woN), "+v"(xoL), "+v"(rdW), "+v"(rdN), "+v"(xr), "+v"(paL), "+v"(prL), "+v"(ooL));
  asm volatile("" : "+s"(hv), "+s"(pv), "+s"(ev), "+s"(job));
  u64 tstart = 0;
  if (j >= 16 || (job >> 16)) tstart = __builtin_amdgcn_s_memrealtime() - 35;
  auto mk64 = [](int lo, int hi) { return (unsigned)lo | ((u64)(unsigned)hi << 32); };
  const u64 ax = mk64(hv[0], hv[1]), aw = mk64(hv[2], hv[3]), myws = mk64(hv[4], hv[5]), asx = mk64(hv[6], hv[7]), asw = mk64(hv[8], hv[9]);
  const int rank = hv[10], pf = hv[11] & 15, dpf = (hv[11] >> 16) & 255, nraw = hv[13], npk = hv[14];
  const u64 dout = mk64(pv.x, pv.y), dws = mk64(pv.z, pv.w), myout = mk64(ev.x, ev.y);
  const u64 mycall = myws + WS_CALL + b * 128;
  const int dest = (rank + 1 + xcd) & 7;
  const int n = job & 15, g0 = (job >> 4) & 63, pnl = (job >> 10) & 3, tgo = (job >> 16) * 10;
  const bool pk = (job >> 12) & 1, foreign = local && n == 7, wide = !((job >> 14) & 1);
  const int nv = wide ? 2 * n : n;
  const bool late = j >= 16, tail = (job >> 13) & 1;
  const int c0 = pnl * 128 + ((job >> 15) & 1) * 64, n0 = dest * NS + c0, r0 = g0 * 16;
  const unsigned lds0 = (unsigned)(size_t)(LDSP unsigned char*)smem, reg = lds0 + 4 * STG + w * 256;
  const i32x4 wsrd = make_srd(aw, KD * ND);
  auto until = [&](int ticks) { while ((int)(unsigned)(__builtin_amdgcn_s_memrealtime() - tstart) < ticks) __builtin_amdgcn_s_sleep(1); };
  unsigned call;
  float scale;
  auto get_call = [&]() {
    unsigned d;
    asm volatile("s_load_dword %0, %1, 0x0\n\ts_waitcnt lgkmcnt(0)" : "=&s"(d) : "s"(mycall) : "memory");
    call = d + 1u;
  };
  bool lost = false;

  if (j >= 16) {
    const i32x4 xall = make_srd(ax, MS * KD);
    auto pf_w = [&](int p, int c) {
      dma_touch(wsrd, (unsigned)(c * 64 + l) * ND + dest * NS + p * 128, 0, reg);
    };
    auto pf_x = [&](int kb, int c) {
      dma_touch(xall, (unsigned)(c * 16 + (l & 15)) * KD + (l >> 4) * 128 + kb * 512, 0, reg);
    };
    const int hw = (j - 16) * 4 + w;
    if (j == 27) {
      u64 pc;
      asm volatile("s_getpc_b64 %0" : "=s"(pc));
      dma_touch(make_srd(pc & ~127ull, CODE_BYTES), (w * 64 + l) * 128, 0, reg);
    }
    if (local) {
      if (pf & 4) {
        until(dpf * 10);
#pragma unroll
        for (int kb = 0; kb < 8; ++kb) { if (hw < 32) pf_w(hw & 3, kb * 8 + (hw >> 2)); else pf_x(kb, hw - 32); }
      }
      get_call();
      lost = wait_counters(myws, WS_XCNT, call * 4u, call, rank, l);
    } else if (j >= 28) {
      u32x4 v[16];
      const unsigned poff = (hw - 48) * KD + l * 16;
      auto ld = [&](int i) {
        asm volatile("buffer_load_dwordx4 %0, %1, %2, %3 offen offset:%4" : "=&v"(v[i]) : "v"(poff), "s"(xall), "s"((unsigned)(448 + (i >> 2) * 16) * KD), "n"((i & 3) * 1024) : "memory");
      };
#pragma unroll
      for (int i = 0; i < 6; ++i) ld(i);
      get_call();
      const i32x4 psrd = make_srd(dws + WS_XS + (call & 1) * XS_PAR + xcd * XS_SRC, 64 * KD);
#pragma unroll
      for (int i = 0; i < 16; ++i) {
        const int younger = i < 6 ? 5 + i : i <= 10 ? 11 : 21 - i;
        switch (younger) {
          case 5: WAITV(5); break; case 6: WAITV(6); break; case 7: WAITV(7); break; case 8: WAITV(8); break;
          case 9: WAITV(9); break; case 10: WAITV(10); break; default: WAITV(11); break;
        }
        asm volatile("" : "+v"(v[i]));
        if (i + 6 < 16) ld(i + 6);
        asm volatile("buffer_store_dwordx4 %0, %1, %2, %3 offen offset:%4 sc0 sc1\n\ts_nop 1" ::"v"(v[i]), "v"(poff), "s"(psrd), "s"((i >> 2) * 16 * KD), "n"((i & 3) * 1024) : "memory");
        if (i == 0) STAMP(8);
      }
      WAITV(0);
      BAR();
      STAMP(9);
      if (threadIdx.x == 0) {
        __hip_atomic_fetch_add((gu32*)(dws + WS_XCNT + rank * 128), 1u, __ATOMIC_RELAXED, __HIP_MEMORY_SCOPE_SYSTEM);
      }
    } else {
      if (pf & 1) {
        const i32x4 lst = *(const CONSTP i32x4*)(kp + 592 + hw * 16);
        const unsigned offw = l * ND + dest * NS, offx = (l & 15) * KD + (l >> 4) * 128;
#pragma unroll
        for (int i = 0; i < 8; ++i) {
          const unsigned e = ((unsigned)lst[i >> 1] >> ((i & 1) * 16)) & 0xffffu;
          if (e & 0x8000) { if (e != 0xffff) dma_touch(wsrd, offw, (e & 63) * 64 * ND + ((e >> 6) & 3) * 128, reg); }
          else dma_touch(xall, offx, (e & 31) * 16 * KD + (e >> 5) * 512, reg);
        }
      }
      if (pf & 2) {
        until(npk);
        pf_w(2 + (hw & 1), hw >> 1);
      }
    }
    if (!local && j >= 28 && (pf & 2)) {
#pragma unroll
      for (int q = 3; q < 8; ++q) pf_w(2 + (hw & 1), q * 8 + ((hw - 48) >> 1));
    }
  }
  auto row_exp = [&](f32x4 v) {
    unsigned m = __builtin_bit_cast(unsigned, fmaxf(fmaxf(fabsf(v.x), fabsf(v.y)), fmaxf(fabsf(v.z), fabsf(v.w))));
    m = max(m, (unsigned)__builtin_amdgcn_update_dpp(0, (int)m, 0x128, 0xf, 0xf, true));
    m = max(m, (unsigned)__builtin_amdgcn_update_dpp(0, (int)m, 0x124, 0xf, 0xf, true));
    m = max(m, (unsigned)__builtin_amdgcn_update_dpp(0, (int)m, 0x122, 0xf, 0xf, true));
    m = max(m, (unsigned)__builtin_amdgcn_update_dpp(0, (int)m, 0x121, 0xf, 0xf, true));
    return (int)((m >> 23) & 255);
  };
  auto enc = [&](f32x4 v, int E) {
    const f32x4 t = {__builtin_amdgcn_ldexpf(v.x, 1 - E), __builtin_amdgcn_ldexpf(v.y, 1 - E), __builtin_amdgcn_ldexpf(v.z, 1 - E), __builtin_amdgcn_ldexpf(v.w, 1 - E)};
    return __builtin_bit_cast(u32x4, t) & 0x80ffffffu;
  };
  auto dec = [&](u32x4 y, int E) {
    const f32x4 t = __builtin_bit_cast(f32x4, y);
    return f32x4{__builtin_amdgcn_ldexpf(t.x, E - 1), __builtin_amdgcn_ldexpf(t.y, E - 1), __builtin_amdgcn_ldexpf(t.z, E - 1), __builtin_amdgcn_ldexpf(t.w, E - 1)};
  };

  const unsigned poff = w * 4096 + l * 16;
  if (n) {
  if (tgo) until(tgo);
  STAMP(1);

  Lane L;
  const unsigned gs = foreign ? XS_SRC : 16 * KD;
  L.xo = xoL + (w >> 1) * gs;
  L.wo[0] = wide ? woW : woN;
  L.wo[1] = wide ? woW + 8 * ND - 1024 : (woN ^ 32) + 16 * ND - 1024;
  L.wo[2] = (woW ^ 64) + 16 * ND - 2048; L.wo[3] = (woW ^ 64) + 24 * ND - 3072;
  L.mw = lds0 + (w << (wide ? 12 : 11)); L.mx = lds0 + 16384 + w * 1024;
  L.skip = w >> 1;
  {
    LDSP unsigned char* const base = (LDSP unsigned char*)smem;
    const int rd = wide ? rdW : rdN, pa = paL ^ (wide ? (w >> 1) * 4096 + (w & 1) * 128 : w * 64);
#pragma unroll
    for (int hf = 0; hf < 2; ++hf) {
      int a0 = rd + hf * 2 * STG, a1 = a0 ^ 16, a2 = xr + hf * 2 * STG, a3 = a2 ^ 64;
      asm("" : "+v"(a0), "+v"(a1), "+v"(a2), "+v"(a3));
      L.rd[hf][0] = base + a0; L.rd[hf][1] = base + a1; L.xr[hf][0] = base + a2; L.xr[hf][1] = base + a3;
    }
    L.pw[0] = base + pa; L.pw[1] = base + (pa ^ 64);
    L.pr = base + prL;
  }
  const unsigned ooff = ooL + c0 * 4;
  const i32x4 osrd = make_srd(dout, NR * MS * NS * 4);

#define CYARG
  f32x4 res[16];
  {
    unsigned par = 0;
    if (foreign) par = call & 1;
    const i32x4 wsrd = make_srd(aw + n0, KD * ND - n0);
    const u64 xown = ax + (unsigned)r0 * KD;
    const i32x4 xs = make_srd(xown + (foreign ? myws + (WS_XS + par * XS_PAR + (g0 - 28) * 16 * KD) - xown : 0ull), foreign ? 7 * XS_SRC : n * 16 * KD);
    const unsigned row0 = (unsigned)(rank * MS + r0) * (NS * 4);
    auto quick = [&](int cnt, int cw) {
      STAMP(11); STAMP(3);
#pragma unroll
      for (int t = 0; t < cnt; ++t) store16(osrd, ooff, row0 + (t / cw) * 16 * NS * 4 + (t % cw) * 256, __builtin_bit_cast(u32x4, res[t] * scale));
    };
#define JOB(N, CW) tile_job<N, CW>(L, wsrd, xs, gs, res, asx, asw, mycall, scale, call CYARG)
    if (!wide) {
      if (n == 4) { JOB(4, 1); quick(4, 1); }
      else JOB(5, 1);
    } else {
      if (n == 2) { JOB(2, 2); quick(4, 2); }
      else if (n == 3) JOB(3, 2);
      else if (n == 5) JOB(5, 2);
      else if (n == 6) JOB(6, 2);
      else if (n == 7) JOB(7, 2);
      else JOB(8, 2);
    }
  }
  if (nv > 4) {
  if (lost) scale = __builtin_nanf("");
  STAMP(11);
  auto voff = [&](int t) { return wide ? (t >> 1) * 16 * NS * 4 + (t & 1) * 256 : t * 16 * NS * 4; };
  auto orow = [&](int t) { return (unsigned)(rank * MS + r0) * (NS * 4) + voff(t); };
  STAMP(3);
  const int units = pk ? nv / 5 : 0;
  auto rest = [&]() {
#pragma unroll
    for (int t = 0; t < 16; ++t) {
      if (t < nv && t >= 5 * units) {
        store16(osrd, ooff, orow(t), __builtin_bit_cast(u32x4, res[t] * scale));
        LSTAMP(t);
      }
    }
  };
  if (pk && nv == 6) gstore16<0>(dout + orow(5), ooff, __builtin_bit_cast(u32x4, res[5] * scale));
  if (pk) {
    const u64 ps = dws + (WS_PK + (call & 1) * PK_PAR + rank * PK_SRC + j * PK_WG);
#pragma unroll
    for (int u = 0; u < 2; ++u) {
      if (u < units) {
        int E[5];
        E[0] = row_exp(res[5 * u]);
        const u32x4 y0 = enc(res[5 * u], E[0]);
        u32x4 hid = (y0 & 0x00ffffffu) | ((y0 >> 7) & 0x01000000u);
#pragma unroll
        for (int g = 1; g < 4; ++g) {
          E[g] = row_exp(res[5 * u + g]);
          const u32x4 v = enc(res[5 * u + g], E[g]) | (((hid >> (7 * (g - 1))) & 127u) << 24);
          if (late) WAITV((u ? WINB : WIN) - 1); else WAITV(WIN1 - 1);
          if (g == 1) gstore16<0>(ps + u * PK_RUN, poff, v); else if (g == 2) gstore16<1024>(ps + u * PK_RUN, poff, v); else gstore16<2048>(ps + u * PK_RUN, poff, v);
          LSTAMP(5 * u + g);
        }
        E[4] = row_exp(res[5 * u + 4]);
        unsigned side = 0;
#pragma unroll
        for (int g = 0; g < 5; ++g) {
          const u32x4 d = __builtin_bit_cast(u32x4, dec(enc(res[5 * u + g], E[g]), E[g])) ^ __builtin_bit_cast(u32x4, res[5 * u + g]);
          side |= ((d.x | d.y | d.z | d.w) ? 1u : 0u) << g;
          side |= ((l >> 1) & 7) == g ? ((E[g] >> ((l & 1) * 4)) & 15) << 8 : 0;
        }
        hid.x |= (side & 7) << 25; hid.y |= ((side >> 3) & 7) << 25; hid.z |= ((side >> 6) & 7) << 25; hid.w |= (side >> 9) << 25;
        if (late) WAITV((u ? WINB : WIN) - 1); else WAITV(WIN1 - 1);
        gstore16<3072>(ps + u * PK_RUN, poff, enc(res[5 * u + 4], E[4]) | ((hid >> 21) << 24));
        LSTAMP(5 * u + 4);
        if (__builtin_amdgcn_ballot_w64(side & 31)) {
#pragma unroll
          for (int g = 0; g < 5; ++g) {
            if ((side >> g) & 1) gstore16<0>(dout + orow(5 * u + g), ooff, __builtin_bit_cast(u32x4, res[5 * u + g] * scale));
          }
        }
      }
    }
    const u64 flag = dws + WS_PKF + ((rank * 32 + j) * 4 + w) * 128;
    if (tail) {
      if (late) WAITV(WINC - 1);
      gstore16<0>(dout + orow(10), ooff, __builtin_bit_cast(u32x4, res[10] * scale));
      gstore16<0>(dout + orow(11), ooff, __builtin_bit_cast(u32x4, res[11] * scale));
      asm volatile("s_waitcnt vmcnt(2)\n\tglobal_store_dword %0, %1, %2 sc0 sc1\n\ts_waitcnt vmcnt(1)" ::"v"(0), "v"(call), "s"(flag) : "memory");
    } else {
      asm volatile("s_waitcnt vmcnt(0)\n\tglobal_store_dword %0, %1, %2 sc0 sc1" ::"v"(0), "v"(call), "s"(flag) : "memory");
    }
    STAMP(2);
  } else if (local) {
    const unsigned base = (unsigned)r0 * (NS * 4), own = (unsigned)rank * (MS * NS * 4);
#pragma unroll
    for (int t = 0; t < 16; ++t) {
      const unsigned o = foreign ? (t < 14 ? (unsigned)((rank + 7 - (t >> 1)) & 7) * (MS * NS * 4) : OOB) : own + (t >> 1) * 16 * NS * 4;
      store16(osrd, ooff, base + o + (t & 1) * 256, __builtin_bit_cast(u32x4, res[t] * scale));
    }
  } else {
    rest();
  }
  }
  STAMP(4);
  if (!pk && !local) WAITV(0);
  BAR();
  STAMP(5);
  if (threadIdx.x == 0) {
    if (tail && !local) __hip_atomic_fetch_add((gu32*)(dws + WS_OCNT + rank * 128), 1u, __ATOMIC_RELAXED, __HIP_MEMORY_SCOPE_SYSTEM);
    __hip_atomic_store((gu32*)mycall, call, __ATOMIC_RELAXED, __HIP_MEMORY_SCOPE_AGENT);
  }
  if (local && j == 0 && w == 2) {
    if (wait_counters(myws, WS_OCNT, call * nraw, call, rank, l) && l == 0)
      ((gf32*)dout)[(size_t)(rank * MS + r0 + 8) * NS + c0] = __builtin_nanf("");
  }
  } else {
    float sx, sw; unsigned d;
    asm volatile("s_load_dword %0, %3, 0x0\n\ts_load_dword %1, %4, 0x0\n\ts_load_dword %2, %5, 0x0\n\ts_waitcnt lgkmcnt(0)"
                 : "=&s"(sx), "=&s"(sw), "=&s"(d) : "s"(asx), "s"(asw), "s"(mycall) : "memory");
    call = d + 1u; scale = sx * sw;
    WAITV(0);
    BAR();
    if (threadIdx.x == 0) __hip_atomic_store((gu32*)mycall, call, __ATOMIC_RELAXED, __HIP_MEMORY_SCOPE_AGENT);
  }
  if (!local) {
    const int src = (rank + 7 - xcd) & 7;
    gu32* tmo = (gu32*)(myws + WS_TMO);
#pragma clang loop unroll(disable)
    for (unsigned list = *(const CONSTP unsigned*)(kp + 208 + j * 4); list & 0x80; list >>= 8) {
      const int js = (list >> 1) & 31, u = list & 1;
      const unsigned jw = *(const CONSTP unsigned*)(kp + 336 + js * 4);
      const bool uw = !((jw >> 14) & 1);
      const i32x4 isrd = make_srd(myws + (WS_PK + (call & 1) * PK_PAR + src * PK_SRC + js * PK_WG + u * PK_RUN), PK_RUN);
      const i32x4 msrd = make_srd(myout + ((unsigned)(src * MS + ((jw >> 4) & 63) * 16) * NS + ((jw >> 10) & 3) * 128 + ((jw >> 15) & 1) * 64) * 4, 128 * NS * 4);
      unsigned vo[5];
#pragma unroll
      for (int g = 0; g < 5; ++g) { const int t = 5 * u + g; vo[g] = uw ? (t >> 1) * 16 * NS * 4 + (t & 1) * 256 : t * 16 * NS * 4; }
      gu32* f = (gu32*)(myws + WS_PKF + ((src * 32 + js) * 4 + w) * 128);
      const u64 t0 = __builtin_amdgcn_s_memrealtime();
      bool bad = false;
      for (unsigned spin = 1; __hip_atomic_load(f, __ATOMIC_RELAXED, __HIP_MEMORY_SCOPE_SYSTEM) < call; ++spin) {
        if ((spin & 15) == 0 && (__hip_atomic_load(tmo, __ATOMIC_RELAXED, __HIP_MEMORY_SCOPE_AGENT) == call || __builtin_amdgcn_s_memrealtime() - t0 > DEADLINE)) {
          __hip_atomic_store(tmo, call, __ATOMIC_RELAXED, __HIP_MEMORY_SCOPE_AGENT);
          bad = true;
          break;
        }
        __builtin_amdgcn_s_sleep(PSLEEP);
      }
      STAMP(12);
      const float sc = bad ? __builtin_nanf("") : scale;
      u32x4 c[4];
      asm volatile("buffer_load_dwordx4 %0, %4, %5, 0 offen sc0 sc1\n\tbuffer_load_dwordx4 %1, %4, %5, 0 offen offset:1024 sc0 sc1\n\t"
                   "buffer_load_dwordx4 %2, %4, %5, 0 offen offset:2048 sc0 sc1\n\tbuffer_load_dwordx4 %3, %4, %5, 0 offen offset:3072 sc0 sc1\n\ts_waitcnt vmcnt(0)"
                   : "=&v"(c[0]), "=&v"(c[1]), "=&v"(c[2]), "=&v"(c[3]) : "v"(poff), "s"(isrd) : "memory");
      const int side = bad ? 0 : (int)(((c[3].x >> 28) & 7) | (((c[3].y >> 28) & 7) << 3) | (((c[3].z >> 28) & 7) << 6) | (((c[3].w >> 28) & 7) << 9));
      const int nib = side >> 8;
      auto put = [&](int g, u32x4 y, int E) { store16(msrd, ((side >> g) & 1) ? OOB : ooL, vo[g], __builtin_bit_cast(u32x4, dec(y, E) * sc)); };
#define PUT(g, y) put(g, y, __builtin_amdgcn_update_dpp(0, nib, 0x150 + 2 * g, 0xf, 0xf, false) | (__builtin_amdgcn_update_dpp(0, nib, 0x151 + 2 * g, 0xf, 0xf, false) << 4))
      PUT(1, c[0] & 0x80ffffffu); PUT(2, c[1] & 0x80ffffffu); PUT(3, c[2] & 0x80ffffffu); PUT(4, c[3] & 0x80ffffffu);
      u32x4 h = ((c[3] >> 3) & 0x00e00000u) | ((c[3] << 4) & 0x80000000u);
#pragma unroll
      for (int k = 0; k < 3; ++k) h |= ((c[k] >> 24) & 127u) << (7 * k);
      PUT(0, h);
    }
  }
}

extern "C" void kernel_launch(int rank, int nranks, ncclComm_t comm, hipStream_t stream, void* const* d_in, const int* in_sizes, int n_in,
                              void* d_out, int out_size, const int* in_dtypes, int out_dtype, void* const* d_out_all, void* const* d_ws_all, size_t ws_bytes) {
  static bool init = false;
  if (!init) { (void)hipFuncSetAttribute((const void*)gemm_a2a, hipFuncAttributeMaxDynamicSharedMemorySize, LDS_BYTES); init = true; }
  Args a;
  for (int i = 0; i < NR; ++i) { int p = (rank + 1 + i) & 7; a.peer[i].out = d_out_all[p]; a.peer[i].ws = d_ws_all[p]; }
  a.x = d_in[0]; a.w = d_in[1]; a.sx = d_in[2]; a.sw = d_in[3];
  a.rank = rank; a.myws = d_ws_all[rank]; a.myout = d_out; a.pad0 = a.pad1 = a.pad2 = 0;
  auto env = [](const char* n, int d) { return d; };
  a.cfg = env("PF", 7) | (env("WIN", 6) << 4) | (env("TWIN", 6) << 8) | (env("DPF", 20) << 16);
  const char* plan = "a4b4A2B2A3B3A3B3A5B5A5B5A6B6C5D5C5D5C6D6C6D6C6D6";
  const int dl = env("DL", 30), pkon = env("PK", 1);
  const double tref = env("TREF", 60) * 0.1, marg = env("MARG", 10) * 0.1, rate = env("RATE", 134) * 0.1, da = env("DA", 50) * 0.1, db = env("DB", 68) * 0.01;
  const int tmax = env("TMAX", 78), tmin2 = env("TMIN2", 40);
  a.nraw = a.npk = 0;
  for (int j = 0; j < 64; ++j) a.job[j] = 0;
  int j = 0, g[4] = {0, 0, 0, 0}, cum = 0, t2 = 0;
  auto add = [&](int n, int p, int narrow, int half, int raw) {
    const int nv = narrow ? n : 2 * n, pk = pkon && !raw && nv >= 5, tail = !pk || nv > 10;
    int t = (int)(10 * (tref + cum / rate - marg - da - db * n));
    if (t < 3) t = 0;
    if (t > tmax) t = tmax;
    if (p >= 2 && j < 16) j = 16;
    if (j >= 16 && t < tmin2) t = tmin2;
    if (p >= 2 && !t2) t2 = t;
    cum += nv - (pk ? nv / 5 : 0);
    a.nraw += tail;
    a.job[j++] = n | (g[p] << 4) | (p << 10) | (pk << 12) | (tail << 13) | (narrow << 14) | (half << 15) | (t << 16);
  };
  for (const char* c = plan; *c; ++c) {
    const int narrow = *c >= 'a', p = (*c - 'A') & 3;
    const int raw = c[1] == 'r';
    c += 1 + raw;
    if (j + narrow >= 28) { fprintf(stderr, "bad plan\n"); return; }
    add(*c - '0', p, narrow, 0, raw);
    if (narrow) add(*c - '0', p, 1, 1, raw);
    g[p] += *c - '0';
  }
  if (g[0] != 28 || g[1] != 28 || g[2] != 28 || g[3] != 28) { fprintf(stderr, "bad plan\n"); return; }
  a.npk = t2 * 10 - env("LEAD", 80);
  for (j = 0; j < 32; ++j) a.job[32 + j] = j < 16 ? (8 | ((8 * (j >> 2)) << 4) | ((j & 3) << 10) | 8192 | (dl << 16)) : (7 | ((28 + ((j - 16) >> 2)) << 4) | ((j & 3) << 10) | 8192);
  a.hold = 0;
  {
    struct Ch { double t; unsigned short e; } ch[352];
    int nc = 0;
    auto due = [&](int q, double frac) { const int n = a.job[q] & 15; return (a.job[q] >> 16) * 0.1 + frac * ((a.job[q] >> 14 & 1) ? 2.1 + 0.49 * n : 3.9 + 0.49 * n); };
    for (int p = 0; p < 2; ++p)
      for (int c = 0; c < 64; ++c) {
        double t = 1e9;
        for (int q = 0; q < 16; ++q) if ((a.job[q] & 15) && (int)(a.job[q] >> 10 & 3) == p && due(q, c / 64.0) < t) t = due(q, c / 64.0);
        ch[nc++] = {t, (unsigned short)(0x8000 | (p << 6) | c)};
      }
    for (int r = 0; r < 28; ++r)
      for (int kb = 0; kb < 8; ++kb) {
        double t = 1e9;
        for (int q = 0; q < 16; ++q) {
          const int n = a.job[q] & 15, g0 = a.job[q] >> 4 & 63;
          if (r >= g0 && r < g0 + n && due(q, kb / 8.0) < t) t = due(q, kb / 8.0);
        }
        ch[nc++] = {t, (unsigned short)((kb << 5) | r)};
      }
    for (int i = 1; i < nc; ++i) { Ch x = ch[i]; int k = i; for (; k > 0 && ch[k - 1].t > x.t; --k) ch[k] = ch[k - 1]; ch[k] = x; }
    for (int i = 0; i < 384; ++i) a.pfl[47 - i % 48][i / 48] = i < nc ? ch[i].e : 0xffff;
  }
  int cnt[32] = {0}, base[32] = {0};
  for (int q = 0; q < 32; ++q) a.unp[q] = 0;
  for (int q = 0; q < 28; ++q) {
    if (q == 16) for (int r = 0; r < 32; ++r) base[r] = cnt[r];
    if (!(a.job[q] >> 12 & 1)) continue;
    const int n = a.job[q] & 15, nv = (a.job[q] >> 14 & 1) ? n : 2 * n;
    for (int u = 0; u < nv / 5; ++u) {
      int t = -1;
      for (int r = 31; r >= 0; --r)
        if (cnt[r] < 4 && ((a.job[r] & 15) == 0 || (q >= 16 ? r < 16 : r < 8 && r + 4 <= q)) && (t < 0 || cnt[r] - base[r] < cnt[t] - base[t])) t = r;
      if (t < 0 || cnt[t] >= 4) { fprintf(stderr, "bad unpack list\n"); return; }
      a.unp[t] |= (unsigned)(0x80 | (q << 1) | u) << (8 * cnt[t]++);
    }
  }
  gemm_a2a<<<256, 256, LDS_BYTES, stream>>>(a);
}
```

```cpp
#include <hip/hip_runtime.h>
#include <rccl/rccl.h>
#include <cstdio>
#include <cstdlib>

typedef unsigned u32x4 __attribute__((ext_vector_type(4)));
typedef int i32x2 __attribute__((ext_vector_type(2)));
typedef int i32x4 __attribute__((ext_vector_type(4)));
typedef int i32x8 __attribute__((ext_vector_type(8)));
typedef int i32x16 __attribute__((ext_vector_type(16)));
typedef float f32x4 __attribute__((ext_vector_type(4)));
typedef unsigned long long u64;
#define LDSP __attribute__((address_space(3)))
#define CONSTP __attribute__((address_space(4)))
typedef __attribute__((address_space(1))) unsigned gu32;
typedef __attribute__((address_space(1))) float gf32;
typedef __attribute__((address_space(1))) u64 gu64;

constexpr int NR = 8, MS = 512, KD = 4096, ND = 4096, NS = 512;
constexpr int STG = 32768;
constexpr int LDS_BYTES = 4 * STG + 1024;
constexpr u64 DEADLINE = 20000000ull;
constexpr unsigned WS_TMO = 128, WS_OCNT = 4096, WS_XCNT = 8192, WS_ENT = 12288, WS_CALL = 65536, WS_PKF = 131072, WS_XS = 524288;
constexpr unsigned XS_SRC = 64 * KD, XS_PAR = NR * XS_SRC;
constexpr unsigned WS_PK = 8u << 20, PK_RUN = 12288, PK_SIDE = 3 * PK_RUN, PK_WG = 40960, PK_SRC = 32 * PK_WG, PK_PAR = NR * PK_SRC;
constexpr unsigned OOB = 0x40000000u;
#define TLEFT 6
#define WIN 4
#define FT 0
#define CNT 1
#define WINB 4
#define WINC 4
#define WIN1 9
#define PSLEEP 1
#define TWIN 8
#define CODE_BYTES 30720
#define STAMP(i)
#define LSTAMP(i)

struct Args {
  const void* x; const void* w; void* myws; const void* sx; const void* sw;
  int rank, cfg, hold, nraw, npk, pad0;
  struct { void* out; void* ws; } peer[NR];
  void* myout; int pad1, pad2;
  unsigned unp[32];
  unsigned job[64];
  unsigned char pfl[32][32];
};

#define WAITV(n) asm volatile("s_waitcnt vmcnt(%0)" ::"n"(n) : "memory")
#define WAITL(n) asm volatile("s_waitcnt lgkmcnt(%0)" ::"n"(n) : "memory")
#define SB() __builtin_amdgcn_sched_barrier(0)
#define BAR() do { __builtin_amdgcn_s_barrier(); asm volatile("" ::: "memory"); } while (0)

template <int LOFF, int OFF>
__device__ __forceinline__ void dma_w(i32x4 srd, unsigned voff, unsigned soff, unsigned lds) {
  asm volatile("s_add_u32 m0, %4, %5\n\ts_nop 0\n\tbuffer_load_dwordx4 %0, %1, %2 offen offset:%3 lds" ::"v"(voff), "s"(srd), "s"(soff), "n"(OFF), "s"(lds), "n"(LOFF) : "memory", "scc");
}
template <int LOFF>
__device__ __forceinline__ void dma_x(i32x4 srd, unsigned voff, unsigned soff, unsigned lds) {
  asm volatile("s_add_u32 m0, %3, %4\n\ts_nop 0\n\tbuffer_load_dwordx4 %0, %1, %2 offen sc0 sc1 lds" ::"v"(voff), "s"(srd), "s"(soff), "s"(lds), "n"(LOFF) : "memory", "scc");
}
template <int LOFF>
__device__ __forceinline__ void dma_xs(i32x4 srd, unsigned voff, unsigned soff, unsigned lds, int skip) {
  asm volatile("s_add_u32 m0, %3, %4\n\ts_cmp_lg_u32 %5, 0\n\ts_cbranch_scc1 1f\n\tbuffer_load_dwordx4 %0, %1, %2 offen sc0 sc1 lds\n1:"
               ::"v"(voff), "s"(srd), "s"(soff), "s"(lds), "n"(LOFF), "s"(skip) : "memory", "scc");
}
__device__ __forceinline__ void dma_touch(i32x4 srd, unsigned voff, unsigned soff, unsigned lds) {
  asm volatile("s_mov_b32 m0, %3\n\ts_nop 0\n\tbuffer_load_dword %0, %1, %2 offen lds" ::"v"(voff), "s"(srd), "s"(soff), "s"(lds) : "memory");
}
__device__ __forceinline__ i32x4 make_srd(u64 a, unsigned bytes) {
  i32x4 r; r.x = (int)(unsigned)a; r.y = (int)((unsigned)(a >> 32) & 0xffffu); r.z = (int)bytes; r.w = 0x00020000; return r;
}
__device__ __forceinline__ void store16(i32x4 srd, unsigned voff, unsigned soff, u32x4 v) {
  asm volatile("buffer_store_dwordx4 %0, %1, %2, %3 offen sc0 sc1\n\ts_nop 1" ::"v"(v), "v"(voff), "s"(srd), "s"(soff) : "memory");
}
template <int OFF>
__device__ __forceinline__ void gstore16(u64 base, unsigned voff, u32x4 v) {
  asm volatile("global_store_dwordx4 %0, %1, %2 offset:%3 sc0 sc1\n\ts_nop 1" ::"v"(voff), "v"(v), "s"(base), "n"(OFF) : "memory");
}
__device__ __forceinline__ bool wait_counters(u64 myws, unsigned off, unsigned target, unsigned call, bool mine, int l) {
  bool bad = false;
  if (mine) {
    gu32* f = (gu32*)(myws + off + l * 128);
    gu32* tmo = (gu32*)(myws + WS_TMO);
    const u64 t0 = __builtin_amdgcn_s_memrealtime();
    for (unsigned spin = 1; __hip_atomic_load(f, __ATOMIC_RELAXED, __HIP_MEMORY_SCOPE_SYSTEM) < target; ++spin) {
      if ((spin & 15) == 0 && (__hip_atomic_load(tmo, __ATOMIC_RELAXED, __HIP_MEMORY_SCOPE_AGENT) == call || __builtin_amdgcn_s_memrealtime() - t0 > DEADLINE)) {
        __hip_atomic_store(tmo, call, __ATOMIC_RELAXED, __HIP_MEMORY_SCOPE_AGENT);
        bad = true;
        break;
      }
      __builtin_amdgcn_s_sleep(1);
    }
  }
  return __builtin_amdgcn_ballot_w64(bad) != 0;
}

struct Lane {
  unsigned xo, wo[4];
  unsigned mw, mx;
  int skip;
  LDSP unsigned char *rd[2][2], *xr[2][2], *pw[2], *pr;
};
template <int I> struct IC { static constexpr int v = I; };

template <int N, int CW>
__device__ __forceinline__ void tile_job(const Lane& L, i32x4 wsrd, i32x4 xsrd, unsigned gs, u64 asx, u64 asw, u64 acall, u64 aseen, float& scale, unsigned& call, unsigned& seen
) {
  constexpr int NW = 2 * CW, NX = (N + 1) / 2, D = NW + NX, DM = NW + N / 2, M = CW * N;
  f32x4 acc[N][CW];
  i32x4 wf[4][CW][2], xf[4][N][2];
  auto dma = [&](auto slot, auto ic, unsigned st) {
    constexpr int i = decltype(ic)::v, so = decltype(slot)::v * STG;
    if constexpr (i < NW) dma_w<so, i * 1024>(wsrd, L.wo[i], st * (128 * ND), L.mw);
    else if constexpr ((N & 1) && i == D - 1) dma_xs<so + (i - NW) * 4096>(xsrd, L.xo, 2 * gs * (i - NW) + st * 128, L.mx, L.skip);
    else if constexpr (i < D) dma_x<so + (i - NW) * 4096>(xsrd, L.xo, 2 * gs * (i - NW) + st * 128, L.mx);
  };
  auto dman = [&](auto slot, int i, unsigned st) {
    switch (i) {
      case 0: dma(slot, IC<0>{}, st); break; case 1: dma(slot, IC<1>{}, st); break; case 2: dma(slot, IC<2>{}, st); break; case 3: dma(slot, IC<3>{}, st); break;
      case 4: dma(slot, IC<4>{}, st); break; case 5: dma(slot, IC<5>{}, st); break; case 6: dma(slot, IC<6>{}, st); break; default: dma(slot, IC<7>{}, st);
    }
  };
  auto rdw = [&](int slot, int c, int h) {
    LDSP unsigned char* p = L.rd[slot >> 1][c] + (slot & 1) * STG + h * 4096 * CW;
    const i32x2 t0 = __builtin_amdgcn_ds_read_tr8_b64_v2i32((LDSP i32x2*)p);
    const i32x2 t1 = __builtin_amdgcn_ds_read_tr8_b64_v2i32((LDSP i32x2*)(p + 512 * CW));
    wf[slot][c][h] = i32x4{t0.x, t0.y, t1.x, t1.y};
  };
  auto rdx = [&](int slot, int g) {
#pragma unroll
    for (int u = 0; u < 2; ++u) xf[slot][g][u] = *(LDSP i32x4*)(L.xr[slot >> 1][u] + (slot & 1) * STG + g * 2048);
  };
  auto mm = [&](int slot, int g, int c) {
    f32x4 d = __builtin_amdgcn_mfma_scale_f32_16x16x128_f8f6f4(__builtin_shufflevector(wf[slot][c][0], wf[slot][c][1], 0, 1, 2, 3, 4, 5, 6, 7),
                                                               __builtin_shufflevector(xf[slot][g][0], xf[slot][g][1], 0, 1, 2, 3, 4, 5, 6, 7), acc[g][c], 1, 1, 0, 0, 0, 0);
    asm volatile("" : "+a"(d));
    acc[g][c] = d;
  };
  auto pw = [&](int g, int c) { *(LDSP f32x4*)(L.pw[c] + g * 4096 * CW) = acc[g][c]; };
  auto step = [&](auto slot, unsigned st, int left) {
    constexpr int sl = decltype(slot)::v, nx = (sl + 1) & 3;
    if (left >= 2) WAITV(2 * DM); else if (left == 1) WAITV(0);
    if (left >= 1) { __builtin_amdgcn_s_waitcnt(0xC07F); BAR(); }
    SB();
#pragma unroll
    for (int m = 0; m < M; ++m) {
      mm(sl, m / CW, m % CW);
      if (left >= 0) {
#pragma unroll
        for (int i = 0; i < N + CW; ++i) {
          if (m == (M >= 8 && CW == 2 ? i : i / 2)) { if (i < CW) { rdw(nx, i, 0); rdw(nx, i, 1); } else rdx(nx, i - CW); }
        }
      } else if (m >= 2) {
        pw((m - 2) / CW, (m - 2) % CW);
      }
      if (left == 3) {
#pragma unroll
        for (int i = (m * D + M - 1) / M; i < ((m + 1) * D + M - 1) / M; ++i) dman(slot, i, st + 4);
      }
      SB();
    }
  };

#pragma unroll
  for (int i = 0; i < D; ++i) dman(IC<0>{}, i, 0);
#pragma unroll
  for (int i = 0; i < D; ++i) dman(IC<1>{}, i, 1);
#pragma unroll
  for (int i = 0; i < D; ++i) dman(IC<2>{}, i, 2);
#pragma unroll
  for (int g = 0; g < N; ++g)
#pragma unroll
    for (int c = 0; c < CW; ++c) { f32x4 z = {0.f, 0.f, 0.f, 0.f}; asm volatile("" : "+a"(z)); acc[g][c] = z; }
#pragma unroll
  for (int i = 2; i < D; ++i) dman(IC<3>{}, i, 3);
  {
    float sx, sw; unsigned d;
    asm volatile("s_load_dword %0, %3, 0x0\n\ts_load_dword %1, %4, 0x0\n\ts_load_dword %2, %5, 0x0\n\t"
                 "s_add_u32 m0, %9, %11\n\ts_nop 0\n\t"
                 "buffer_load_dwordx4 %6, %8, %10 offen lds\n\tbuffer_load_dwordx4 %7, %8, %10 offen offset:1024 lds\n\t"
                 "s_waitcnt lgkmcnt(0)"
                 : "=&s"(sx), "=&s"(sw), "=&s"(d)
                 : "s"(asx), "s"(asw), "s"(acall), "v"(L.wo[0]), "v"(L.wo[1]), "s"(wsrd), "s"(L.mw), "s"(384 * ND), "n"(3 * STG)
                 : "memory", "scc");
    call = d + 1u; scale = sx * sw;
  }
  WAITV(3 * DM); BAR();
#pragma unroll
  for (int i = 0; i < NW; ++i) rdw(0, i >> 1, i & 1);
#pragma unroll
  for (int g = 0; g < N; ++g) rdx(0, g);
  SB();
#pragma clang loop unroll(disable)
  for (unsigned st = 0; st < 28; st += 4) {
    asm volatile("" : "+s"(st));
    step(IC<0>{}, st, 3); step(IC<1>{}, st + 1, 3); step(IC<2>{}, st + 2, 3); step(IC<3>{}, st + 3, 3);
  }
  step(IC<0>{}, 28, 2);
  if (M > 4) asm volatile("s_load_dword %0, %1, 0x0\n\ts_waitcnt vmcnt(0) lgkmcnt(0)" : "=&s"(seen) : "s"(aseen) : "memory");
  step(IC<1>{}, 29, 1); step(IC<2>{}, 30, 0); step(IC<3>{}, 31, -1);
  pw((M - 2) / CW, (M - 2) % CW); pw(N - 1, CW - 1);
  WAITL(0); BAR();
}

__global__ __launch_bounds__(256, 1) void gemm_a2a(Args args) {
  extern __shared__ __attribute__((aligned(16))) unsigned char smem[];
  const int b = blockIdx.x, xcd = b & 7, j = b >> 3;
  const bool local = xcd == 7;
  const CONSTP unsigned char* kp = (const CONSTP unsigned char*)__builtin_amdgcn_kernarg_segment_ptr();
  i32x16 hv = *(const CONSTP i32x16*)kp;
  i32x4 pv = *(const CONSTP i32x4*)(kp + 64 + xcd * 16);
  i32x2 ev = *(const CONSTP i32x2*)(kp + 192);
  unsigned job = *(const CONSTP unsigned*)(kp + 336 + (unsigned)((local ? 128 : 0) + j * 4));
  const int w = __builtin_amdgcn_readfirstlane(threadIdx.x >> 6), l = threadIdx.x & 63;
  i32x4 lst = *(const CONSTP i32x4*)(kp + 592 + (unsigned)(j * 32));
  SB();
  int woW = (w * 32 + (l >> 3)) * ND + (((l & 7) ^ ((l >> 4) & 3)) << 4), woN = (w * 32 + (l >> 2)) * ND + (((l & 3) ^ ((l >> 4) & 1)) << 4);
  int xoL = ((w & 1) * 8 + (l >> 3)) * KD + (((l & 7) ^ ((l >> 4) & 3) ^ ((w & 1) << 2)) << 4);
  const int kr = 16 * (l >> 4) + ((l & 15) >> 1);
  int rdW = ((kr * 128 + (((l >> 2) & 7) << 4)) ^ (w << 5)) + (l & 1) * 8, rdN = ((kr * 64 + (((l >> 3) & 3) << 4)) ^ (w << 4)) + (l & 1) * 8;
  int xr = 16384 + (l & 15) * 128 + (((l >> 4) ^ ((l >> 1) & 7)) << 4);
  int paL = (l & 15) * 256 + (((l >> 4) ^ (l & 7)) << 4);
  int prL = (w * 4 + (l >> 4)) * 256 + (((l & 15) ^ ((w & 1) * 4 + (l >> 4))) << 4);
  int ooL = ((w * 4 + (l >> 4)) * NS + (l & 15) * 4) * 4;
  asm volatile("" : "+v"(woW), "+v"(woN), "+v"(xoL), "+v"(rdW), "+v"(rdN), "+v"(xr), "+v"(paL), "+v"(prL), "+v"(ooL));
  asm volatile("" : "+s"(hv), "+s"(pv), "+s"(ev), "+s"(job), "+s"(lst));
  u64 tstart = 0;
  if (j >= 16 || ((job >> 16) & 255)) tstart = __builtin_amdgcn_s_memrealtime() - 35;
  auto mk64 = [](int lo, int hi) { return (unsigned)lo | ((u64)(unsigned)hi << 32); };
  const u64 ax = mk64(hv[0], hv[1]), aw = mk64(hv[2], hv[3]), myws = mk64(hv[4], hv[5]), asx = mk64(hv[6], hv[7]), asw = mk64(hv[8], hv[9]);
  const int rank = hv[10], pf = hv[11] & 15, dpf = (hv[11] >> 16) & 255, nraw = hv[13], npk = hv[14];
  const u64 dout = mk64(pv.x, pv.y), dws = mk64(pv.z, pv.w), myout = mk64(ev.x, ev.y);
  const u64 mycall = myws + WS_CALL + b * 128;
  const int dest = (rank + 1 + xcd) & 7;
  const int n = job & 15, g0 = (job >> 4) & 63, pnl = (job >> 10) & 3, tgo = ((job >> 16) & 255) * 10;
  const int units = (job >> 24) & 3;
  const bool pk = units, foreign = local && n == 7, wide = !((job >> 14) & 1);
  const int nv = wide ? 2 * n : n;
  const bool late = pnl >= 2, tail = (job >> 13) & 1;
  const int c0 = pnl * 128 + ((job >> 15) & 1) * 64, n0 = dest * NS + c0, r0 = g0 * 16;
  const unsigned lds0 = (unsigned)(size_t)(LDSP unsigned char*)smem, reg = lds0 + 4 * STG + w * 256;
  const i32x4 wsrd = make_srd(aw, KD * ND);
  auto until = [&](int ticks) {
    for (;;) {
      const int left = ticks - (int)(unsigned)(__builtin_amdgcn_s_memrealtime() - tstart);
      if (left <= 0) break;
      if (left > 40) __builtin_amdgcn_s_sleep(10); else __builtin_amdgcn_s_sleep(1);
    }
  };
  unsigned call;
  float scale;
  auto get_call = [&]() {
    unsigned d;
    asm volatile("s_load_dword %0, %1, 0x0\n\ts_waitcnt lgkmcnt(0)" : "=&s"(d) : "s"(mycall) : "memory");
    call = d + 1u;
  };
  bool lost = false;
  auto prefetch = [&](i32x4 list) {
    const i32x4 xall = make_srd(ax, MS * KD);
    const unsigned offw = (w * 64 + l) * ND + dest * NS, offx = (w * 16 + (l & 15)) * KD + (l >> 4) * 128;
    u64 lo = mk64(list.x, list.y), hi = mk64(list.z, list.w);
#pragma clang loop unroll(disable)
    for (; !(lo & 0x80); lo = (lo >> 8) | (hi << 56), hi = (hi >> 8) | (0xffull << 56)) {
      const unsigned e = (unsigned)lo;
      if (e & 0x40) dma_touch(wsrd, offw, (e & 15) * 256 * ND + ((e >> 4) & 3) * 128, reg);
      else dma_touch(xall, offx, (e & 7) * 64 * KD + ((e >> 3) & 7) * 512, reg);
    }
  };

  if (local) {
    until(dpf * 10);
    const int na = pf >> 1;
    unsigned e[16];
#pragma unroll
    for (int i = 0; i < 16; ++i) {
      const int kb = j < 16 ? (i < na ? 2 * i : 99) : (i < na ? 2 * i + 1 : na + i);
      const int h = (j + 5 * i) & 15;
      e[i] = kb > 7 ? 0xffu : h < 8 ? 0x40u | ((h & 3) << 4) | (2 * kb + (h >> 2)) : (kb << 3) | (h - 8);
    }
    prefetch(i32x4{(int)(e[0] | (e[1] << 8) | (e[2] << 16) | (e[3] << 24)), (int)(e[4] | (e[5] << 8) | (e[6] << 16) | (e[7] << 24)), -1, -1});
  }
  if (j >= 16) {
    const i32x4 xall = make_srd(ax, MS * KD);
    const int hw = (j - 16) * 4 + w;
    if (j == 27) {
      u64 pc;
      asm volatile("s_getpc_b64 %0" : "=s"(pc));
      dma_touch(make_srd(pc & ~127ull, CODE_BYTES), (w * 64 + l) * 128, 0, reg);
    }
    if (local) {
      get_call();
      until(550);
      lost = wait_counters(myws, WS_XCNT, call * 4u, call, l < NR && l != rank, l);
    } else if (j >= 28) {
      u32x4 v[16];
      const unsigned poff = (hw - 48) * KD + l * 16;
      auto ld = [&](int i) {
        asm volatile("buffer_load_dwordx4 %0, %1, %2, %3 offen offset:%4" : "=&v"(v[i]) : "v"(poff), "s"(xall), "s"((unsigned)(448 + (i >> 2) * 16) * KD), "n"((i & 3) * 1024) : "memory");
      };
#pragma unroll
      for (int i = 0; i < 6; ++i) ld(i);
      get_call();
      if (hw == 48) asm volatile("global_store_dword %0, %1, %2 sc0 sc1" ::"v"(0), "v"(call), "s"(dws + WS_ENT + rank * 128) : "memory");
      const i32x4 psrd = make_srd(dws + WS_XS + (call & 1) * XS_PAR + xcd * XS_SRC, 64 * KD);
#pragma unroll
      for (int i = 0; i < 16; ++i) {
        const int younger = i < 6 ? 5 + i : i <= 10 ? 11 : 21 - i;
        switch (younger) {
          case 5: WAITV(5); break; case 6: WAITV(6); break; case 7: WAITV(7); break; case 8: WAITV(8); break;
          case 9: WAITV(9); break; case 10: WAITV(10); break; default: WAITV(11); break;
        }
        asm volatile("" : "+v"(v[i]));
        if (i + 6 < 16) ld(i + 6);
        asm volatile("buffer_store_dwordx4 %0, %1, %2, %3 offen offset:%4 sc0 sc1\n\ts_nop 1" ::"v"(v[i]), "v"(poff), "s"(psrd), "s"((i >> 2) * 16 * KD), "n"((i & 3) * 1024) : "memory");
        if (i == 0) STAMP(8);
      }
      WAITV(0);
      BAR();
      STAMP(9);
      if (threadIdx.x == 0) {
        __hip_atomic_fetch_add((gu32*)(dws + WS_XCNT + rank * 128), 1u, __ATOMIC_RELAXED, __HIP_MEMORY_SCOPE_SYSTEM);
      }
    }
  }
  if (!local) prefetch(lst);
  const unsigned poff = w * 3072 + l * 16;
  if (n) {
  if (tgo) until(tgo);
  STAMP(1);

  Lane L;
  const unsigned gs = foreign ? XS_SRC : 16 * KD;
  L.xo = xoL + (w >> 1) * gs;
  L.wo[0] = wide ? woW : woN;
  L.wo[1] = wide ? woW + 8 * ND - 1024 : (woN ^ 32) + 16 * ND - 1024;
  L.wo[2] = (woW ^ 64) + 16 * ND - 2048; L.wo[3] = (woW ^ 64) + 24 * ND - 3072;
  L.mw = lds0 + (w << (wide ? 12 : 11)); L.mx = lds0 + 16384 + w * 1024;
  L.skip = w >> 1;
  {
    LDSP unsigned char* const base = (LDSP unsigned char*)smem;
    const int rd = wide ? rdW : rdN, pa = paL ^ (wide ? (w >> 1) * 4096 + (w & 1) * 128 : w * 64);
#pragma unroll
    for (int hf = 0; hf < 2; ++hf) {
      int a0 = rd + hf * 2 * STG, a1 = a0 ^ 16, a2 = xr + hf * 2 * STG, a3 = a2 ^ 64;
      asm("" : "+v"(a0), "+v"(a1), "+v"(a2), "+v"(a3));
      L.rd[hf][0] = base + a0; L.rd[hf][1] = base + a1; L.xr[hf][0] = base + a2; L.xr[hf][1] = base + a3;
    }
    L.pw[0] = base + pa; L.pw[1] = base + (pa ^ 64);
    L.pr = base + prL;
  }
  const unsigned ooff = ooL + c0 * 4;
  const i32x4 osrd = make_srd(dout, NR * MS * NS * 4);

#define CYARG
  unsigned seen = 0;
  auto res = [&](int t) { return *(LDSP f32x4*)(L.pr + t * 4096); };
  {
    unsigned par = 0;
    if (foreign) par = call & 1;
    const i32x4 wsrd = make_srd(aw + n0, KD * ND - n0);
    const u64 xown = ax + (unsigned)r0 * KD;
    const i32x4 xs = make_srd(xown + (foreign ? myws + (WS_XS + par * XS_PAR + (g0 - 28) * 16 * KD) - xown : 0ull), foreign ? 7 * XS_SRC : n * 16 * KD);
    const unsigned row0 = (unsigned)(rank * MS + r0) * (NS * 4);
    auto quick = [&](int cnt, int cw) {
      STAMP(11); STAMP(3);
      f32x4 v[4];
#pragma unroll
      for (int t = 0; t < cnt; ++t) v[t] = res(t);
#pragma unroll
      for (int t = 0; t < cnt; ++t) store16(osrd, ooff, row0 + (t / cw) * 16 * NS * 4 + (t % cw) * 256, __builtin_bit_cast(u32x4, v[t] * scale));
    };
#define JOB(N, CW) tile_job<N, CW>(L, wsrd, xs, gs, asx, asw, mycall, myws + WS_ENT + dest * 128, scale, call, seen CYARG)
    if (!wide) {
      if (n == 2) { JOB(2, 1); quick(2, 1); }
      else { JOB(4, 1); quick(4, 1); }
    } else {
      if (n == 2) { JOB(2, 2); quick(4, 2); }
      else if (n == 3) JOB(3, 2);
      else if (n == 4) JOB(4, 2);
      else if (n == 5) JOB(5, 2);
      else if (n == 6) JOB(6, 2);
      else if (n == 7) JOB(7, 2);
      else JOB(8, 2);
    }
  }
  if (nv > 4) {
  if (lost) scale = __builtin_nanf("");
  STAMP(11);
  STAMP(3);
  const u64 orow = dout + (unsigned)(rank * MS + r0) * (NS * 4);
  auto raw2 = [&](int g) {
    const f32x4 a = res(2 * g), c = res(2 * g + 1);
    gstore16<0>(orow + g * (16 * NS * 4), ooff, __builtin_bit_cast(u32x4, a * scale));
    gstore16<256>(orow + g * (16 * NS * 4), ooff, __builtin_bit_cast(u32x4, c * scale));
  };
  if (units) {
    if (n & 1) raw2(n - 1);
    const u64 ps = dws + (WS_PK + (call & 1) * PK_PAR + rank * PK_SRC + j * PK_WG);
    unsigned side = 0;
#pragma clang loop unroll(disable)
    for (int u = 0; u < units; ++u) {
      f32x4 v[4];
      i32x4 q[4];
      unsigned m = 0, esc = 0;
#pragma unroll
      for (int g = 0; g < 4; ++g) v[g] = res(4 * u + g);
      int K[4];
#pragma unroll
      for (int g = 0; g < 4; ++g) {
        float t;
        asm("v_max3_f32 %0, |%1|, |%2|, |%3|" : "=v"(t) : "v"(v[g].x), "v"(v[g].y), "v"(v[g].z));
        asm("v_max_f32 %0, %1, |%2|" : "=v"(K[g]) : "v"(t), "v"(v[g].w));
      }
#define ROR(c) _Pragma("unroll") for (int g = 0; g < 4; ++g) K[g] = max((unsigned)K[g], (unsigned)__builtin_amdgcn_update_dpp(0, K[g], c, 0xf, 0xf, true)); SB();
      SB(); ROR(0x128) ROR(0x124) ROR(0x122) ROR(0x121)
#pragma unroll
      for (int g = 0; g < 4; ++g) {
        const int k = 157 - (((unsigned)K[g] >> 23) & 255);
        unsigned d = 0;
#pragma unroll
        for (int c = 0; c < 4; ++c) {
          const float x = v[g][c], t = __builtin_amdgcn_ldexpf(x, k);
          int r;
          asm("v_cvt_i32_f32 %0, %1" : "=v"(r) : "v"(t));
          q[g][c] = r;
          d |= __builtin_bit_cast(unsigned, (float)r) ^ __builtin_bit_cast(unsigned, t);
        }
        const unsigned low = __builtin_amdgcn_perm(__builtin_amdgcn_perm(q[g].w, q[g].z, 0x0400u), __builtin_amdgcn_perm(q[g].y, q[g].x, 0x0400u), 0x05040100u);
        const bool bad = (d | (low & 0x7f7f7f7fu)) != 0 || q[g].x <= (int)0x800000ffu;
        m |= bad ? 16u << g : low >> (7 - g);
      }
      while (__builtin_amdgcn_ballot_w64(__builtin_popcount(m & 0x0f0f0f0fu) > 2)) {
        const int top = (31 - __builtin_clz((m & 0x0f0f0f0fu) | 1)) & 3;
        m = __builtin_popcount(m & 0x0f0f0f0fu) > 2 ? (m & ~(0x01010101u << top)) | (16u << top) : m;
      }
      esc = (m >> 4) & 15;
#pragma unroll
      for (int g = 0; g < 4; ++g) q[g].x = ((m >> (4 + g)) & 1) ? (int)0x80000000u : q[g].x;
#pragma unroll
      for (int k = 0; k < 3; ++k) {
        u32x4 y;
#pragma unroll
        for (int c = 0; c < 4; ++c) { const int h = q[0][c], z = q[k + 1][c]; y[c] = __builtin_amdgcn_perm(h, z, 0x03020105u + k); }
        if (late) WAITV(WIN - 1);
        if (k == 0) gstore16<0>(ps + u * PK_RUN, poff, y); else if (k == 1) gstore16<1024>(ps + u * PK_RUN, poff, y); else gstore16<2048>(ps + u * PK_RUN, poff, y);
      }
      {
        const unsigned k01 = (l & 1) ? K[1] : K[0], k23 = (l & 1) ? K[3] : K[2], mine = (l & 2) ? k23 : k01;
        side |= ((l >> 2) & 3) == u ? (mine << 1) & 0xff000000u : 0u;
        m &= 0x0f0f0f0fu;
        const unsigned p = m ? __builtin_ctz(m) | ((31 - __builtin_clz(m)) << 8) : 1;
        const unsigned z = (p & 0x0303u) | ((p >> 1) & 0x0c0cu);
        side |= ((z | (z >> 4)) & 255u) << (8 * u);
      }
      if (__builtin_amdgcn_ballot_w64(esc)) {
        if (seen < call) {
          if (wait_counters(myws, WS_ENT, call, call, l == dest, l)) scale = __builtin_nanf("");
          seen = call;
        }
#pragma unroll
        for (int g = 0; g < 4; ++g)
          if ((esc >> g) & 1) gstore16<0>(orow + ((2 * u + (g >> 1)) * 16 * NS * 4 + (g & 1) * 256), ooff, __builtin_bit_cast(u32x4, v[g] * scale));
      }
    }
    asm volatile("global_store_dword %0, %1, %2 sc0 sc1" ::"v"(w * 256 + l * 4), "v"(side), "s"(ps + PK_SIDE) : "memory");
    const u64 flag = dws + WS_PKF + ((rank * 32 + j) * 4 + w) * 128;
    if (tail) {
      WAITV(WINC - 1);
      raw2(4); raw2(5);
      asm volatile("s_waitcnt vmcnt(4)\n\tglobal_store_dword %0, %1, %2 sc0 sc1\n\ts_waitcnt vmcnt(1)" ::"v"(0), "v"(call), "s"(flag) : "memory");
    } else {
      asm volatile("s_waitcnt vmcnt(0)\n\tglobal_store_dword %0, %1, %2 sc0 sc1" ::"v"(0), "v"(call), "s"(flag) : "memory");
    }
    STAMP(2);
  } else {
#pragma clang loop unroll(disable)
    for (int g = 0; g < n; g += 4) {
      f32x4 v[8];
#pragma unroll
      for (int t = 0; t < 8; ++t) v[t] = res(2 * g + t);
#pragma unroll
      for (int t = 0; t < 8; ++t) {
        const int i = g + (t >> 1);
        const unsigned row = foreign ? ((rank + 7 - i) & 7) * MS + r0 : rank * MS + r0 + i * 16;
        store16(osrd, ooff, i < n ? row * (NS * 4) + (t & 1) * 256 : OOB, __builtin_bit_cast(u32x4, v[t] * scale));
      }
    }
  }
  }
  STAMP(4);
  if (!pk && !local) WAITV(0);
  BAR();
  STAMP(5);
  if (threadIdx.x == 0) {
    if (tail && !local) __hip_atomic_fetch_add((gu32*)(dws + WS_OCNT + rank * 128), 1u, __ATOMIC_RELAXED, __HIP_MEMORY_SCOPE_SYSTEM);
    __hip_atomic_store((gu32*)mycall, call, __ATOMIC_RELAXED, __HIP_MEMORY_SCOPE_AGENT);
  }
  if (local && j == 0 && w == 2) {
    if (wait_counters(myws, WS_OCNT, call * nraw, call, l < NR && l != rank, l) && l == 0)
      ((gf32*)dout)[(size_t)(rank * MS + r0 + 8) * NS + c0] = __builtin_nanf("");
  }
  } else {
    float sx, sw; unsigned d;
    asm volatile("s_load_dword %0, %3, 0x0\n\ts_load_dword %1, %4, 0x0\n\ts_load_dword %2, %5, 0x0\n\ts_waitcnt lgkmcnt(0)"
                 : "=&s"(sx), "=&s"(sw), "=&s"(d) : "s"(asx), "s"(asw), "s"(mycall) : "memory");
    call = d + 1u; scale = sx * sw;
    WAITV(0);
    BAR();
    if (threadIdx.x == 0) __hip_atomic_store((gu32*)mycall, call, __ATOMIC_RELAXED, __HIP_MEMORY_SCOPE_AGENT);
  }
  if (!local) { prefetch(*(const CONSTP i32x4*)(kp + 608 + (unsigned)(j * 32))); WAITV(0); }
  if (!local) {
    const int src = (rank + 7 - xcd) & 7;
    gu32* tmo = (gu32*)(myws + WS_TMO);
#pragma clang loop unroll(disable)
    for (unsigned list = *(const CONSTP unsigned*)(kp + 208 + j * 4); list & 0x80; list >>= 8) {
      const int js = (list >> 2) & 31, u = list & 3;
      const unsigned jw = *(const CONSTP unsigned*)(kp + 336 + js * 4);
      const i32x4 isrd = make_srd(myws + (WS_PK + (call & 1) * PK_PAR + src * PK_SRC + js * PK_WG), PK_WG);
      const i32x4 msrd = make_srd(myout + ((unsigned)(src * MS + (((jw >> 4) & 63) + 2 * u) * 16) * NS + ((jw >> 10) & 3) * 128) * 4, 32 * NS * 4);
      gu32* f = (gu32*)(myws + WS_PKF + ((src * 32 + js) * 4 + w) * 128);
      const u64 t0 = __builtin_amdgcn_s_memrealtime();
      bool bad = false;
      for (unsigned spin = 1; __hip_atomic_load(f, __ATOMIC_RELAXED, __HIP_MEMORY_SCOPE_SYSTEM) < call; ++spin) {
        if ((spin & 15) == 0 && (__hip_atomic_load(tmo, __ATOMIC_RELAXED, __HIP_MEMORY_SCOPE_AGENT) == call || __builtin_amdgcn_s_memrealtime() - t0 > DEADLINE)) {
          __hip_atomic_store(tmo, call, __ATOMIC_RELAXED, __HIP_MEMORY_SCOPE_AGENT);
          bad = true;
          break;
        }
        __builtin_amdgcn_s_sleep(PSLEEP);
      }
      STAMP(12);
      const float sc = bad ? __builtin_nanf("") : scale;
      u32x4 c[3];
      unsigned side;
      asm volatile("buffer_load_dword %3, %5, %6, 0 offen sc0 sc1\n\tbuffer_load_dwordx4 %0, %4, %6, %7 offen sc0 sc1\n\t"
                   "buffer_load_dwordx4 %1, %4, %6, %7 offen offset:1024 sc0 sc1\n\tbuffer_load_dwordx4 %2, %4, %6, %7 offen offset:2048 sc0 sc1\n\ts_waitcnt vmcnt(0)"
                   : "=&v"(c[0]), "=&v"(c[1]), "=&v"(c[2]), "=&v"(side) : "v"(poff), "v"(PK_SIDE + w * 256 + l * 4), "s"(isrd), "s"(u * PK_RUN) : "memory");
      const unsigned lo = (side >> (8 * u)) & 15, hi = (side >> (8 * u + 4)) & 15;
      const unsigned m = lo <= hi ? (1u << (lo + (lo & 12))) | (1u << (hi + (hi & 12))) : 0u;
      const int er = __builtin_amdgcn_ds_bpermute(((l & 48) | ((l + 4 * u) & 15)) * 4, (int)(side >> 24) - 157);
      auto put = [&](int g, u32x4 h, int k) {
        const unsigned low = (m << (7 - g)) & 0x80808080u;
        f32x4 o;
#pragma unroll
        for (int i = 0; i < 4; ++i) { const unsigned x = h[i]; o[i] = __builtin_amdgcn_ldexpf((float)(int)__builtin_amdgcn_perm(low, x, 0x03020104u + i), k); }
        store16(msrd, ((h.x >> 8) == 0x800000u && !bad) ? OOB : ooL, (g >> 1) * 16 * NS * 4 + (g & 1) * 256, __builtin_bit_cast(u32x4, o * sc));
      };
#define PUT(g, h) put(g, h, __builtin_amdgcn_update_dpp(0, er, 0x150 + g, 0xf, 0xf, false))
      PUT(1, c[0]); PUT(2, c[1]); PUT(3, c[2]);
      u32x4 h0;
#pragma unroll
      for (int i = 0; i < 4; ++i) { const unsigned x = c[0][i], y = c[1][i], z = c[2][i]; h0[i] = __builtin_amdgcn_perm(z, __builtin_amdgcn_perm(y, x, 0x00040000u), 0x04020100u); }
      PUT(0, h0);
    }
  }
}

extern "C" void kernel_launch(int rank, int nranks, ncclComm_t comm, hipStream_t stream, void* const* d_in, const int* in_sizes, int n_in,
                              void* d_out, int out_size, const int* in_dtypes, int out_dtype, void* const* d_out_all, void* const* d_ws_all, size_t ws_bytes) {
  static bool init = false;
  if (!init) { (void)hipFuncSetAttribute((const void*)gemm_a2a, hipFuncAttributeMaxDynamicSharedMemorySize, LDS_BYTES); init = true; }
  Args a;
  for (int i = 0; i < NR; ++i) { int p = (rank + 1 + i) & 7; a.peer[i].out = d_out_all[p]; a.peer[i].ws = d_ws_all[p]; }
  a.x = d_in[0]; a.w = d_in[1]; a.sx = d_in[2]; a.sw = d_in[3];
  a.rank = rank; a.myws = d_ws_all[rank]; a.myout = d_out; a.pad0 = a.pad1 = a.pad2 = 0;
  auto env = [](const char* n, int d) { return d; };
  a.cfg = (env("NA", 1) << 1) | (env("WIN", 6) << 4) | (env("TWIN", 6) << 8) | (env("DPF", 15) << 16);
  const char* plan = "a4b4A2B2A3B3A3B3A5B5A5B5A6B6C4D4C6D6C6D6C6Dr6Cr6Dr6";
  const int dl = env("DL", 25), pkon = env("PK", 1);
  const double tref = env("TREF", 60) * 0.1, marg = env("MARG", 15) * 0.1, rate = env("RATE", 134) * 0.1, da = env("DA", 50) * 0.1, db = env("DB", 68) * 0.01;
  const int tmax = env("TMAX", 78), tmin2 = env("TMIN2", 33);
  a.nraw = a.npk = 0;
  for (int j = 0; j < 64; ++j) a.job[j] = 0;
  int j = 0, g[4] = {0, 0, 0, 0}, t2 = 0;
  double cum = 0;
  auto add = [&](int n, int p, int narrow, int half, int raw) {
    const int nv = narrow ? n : 2 * n, pk = pkon && nv >= 6 ? (raw ? 2 : nv / 4) : 0, tail = !pk || raw;
    int t = (int)(10 * (tref + cum / rate - marg - da - db * n));
    if (t < 3) t = 0;
    if (t > tmax) t = tmax;
    if (p >= 2 && j < 16) j = 16;
    if (p >= 2 && t < tmin2) t = tmin2;
    if (narrow) t = 0;
    if (p >= 2 && !t2) t2 = t;
    cum += nv - pk + (pk ? 0.25 : 0);
    a.nraw += tail;
    a.job[j++] = n | (g[p] << 4) | (p << 10) | (tail << 13) | (narrow << 14) | (half << 15) | (t << 16) | (pk << 24);
  };
  for (const char* c = plan; *c; ++c) {
    const int narrow = *c >= 'a', p = (*c - 'A') & 3;
    const int raw = c[1] == 'r';
    c += 1 + raw;
    if (j + narrow >= 28) { fprintf(stderr, "bad plan\n"); return; }
    add(*c - '0', p, narrow, 0, raw);
    if (narrow) add(*c - '0', p, 1, 1, raw);
    g[p] += *c - '0';
  }
  if (g[0] != 28 || g[1] != 28 || g[2] != 28 || g[3] != 28) { fprintf(stderr, "bad plan\n"); return; }
  a.npk = t2 * 10 - env("LEAD", 80);
  for (j = 0; j < 32; ++j) a.job[32 + j] = j < 16 ? (8 | ((8 * (j >> 2)) << 4) | ((j & 3) << 10) | 8192 | (dl << 16)) : (7 | ((28 + ((j - 16) >> 2)) << 4) | ((j & 3) << 10) | 8192);
  a.hold = 0;
  {
    struct Ch { double t; unsigned char e; } ch[120];
    int nc = 0;
    auto dur = [&](int q) { const int n = a.job[q] & 15; return (a.job[q] >> 14 & 1) ? 2.9 + 0.35 * n : 3.9 + 0.49 * n; };
    auto due = [&](int q, double frac) { return (a.job[q] >> 16 & 255) * 0.1 + frac * dur(q); };
    for (int p = 0; p < 4; ++p)
      for (int c = 0; c < 16; ++c) {
        double t = 1e9;
        for (int q = 0; q < 28; ++q) if ((a.job[q] & 15) && (int)(a.job[q] >> 10 & 3) == p && due(q, c / 16.0) < t) t = due(q, c / 16.0);
        ch[nc++] = {t, (unsigned char)(0x40 | (p << 4) | c)};
      }
    for (int r = 0; r < 7; ++r)
      for (int kb = 0; kb < 8; ++kb) {
        double t = 1e9;
        for (int q = 0; q < 28; ++q) {
          const int n = a.job[q] & 15, g0 = a.job[q] >> 4 & 63;
          if (4 * r + 3 >= g0 && 4 * r < g0 + n && due(q, kb / 8.0) < t) t = due(q, kb / 8.0);
        }
        ch[nc++] = {t, (unsigned char)((kb << 3) | r)};
      }
    for (int i = 1; i < nc; ++i) { Ch x = ch[i]; int k = i; for (; k > 0 && ch[k - 1].t > x.t; --k) ch[k] = ch[k - 1]; ch[k] = x; }
    const double rtt = env("RTT", 70) * 0.01, tpush = env("TPUSH", 66) * 0.1, tafter = env("TAFT", 33) * 0.1;
    int len[32][2] = {{0}}, k = 0;
    for (int q = 0; q < 32; ++q) for (int i = 0; i < 32; ++i) a.pfl[q][i] = 0xff;
    for (int r = 0; k < nc && r < 40 && (env("PF", 7) & 1); ++r)
      for (int q = 31; q >= 0 && k < nc; --q) {
        const int n = a.job[q] & 15, nv = (a.job[q] >> 14 & 1) ? n : 2 * n;
        const double t0 = 0.4 + r * rtt, st = (a.job[q] >> 16 & 255) * 0.1;
        int h = -1;
        if (q >= 28) { if (t0 >= tpush) h = 0; }
        else if (n == 0 || t0 + rtt <= st) h = 0;
        else if (nv <= 4 && t0 >= st + dur(q) + tafter) h = 1;
        if (h >= 0 && len[q][h] < 16) a.pfl[q][16 * h + len[q][h]++] = ch[k++].e;
      }
  }
  int cnt[32] = {0}, base[32] = {0};
  for (int q = 0; q < 32; ++q) a.unp[q] = 0;
  for (int q = 0; q < 28; ++q) {
    if ((a.job[q] >> 11 & 1) && !(a.job[q - 1] >> 11 & 1)) for (int r = 0; r < 32; ++r) base[r] = cnt[r];
    for (int u = 0; u < (int)(a.job[q] >> 24); ++u) {
      int t = -1;
      for (int r = 31; r >= 0; --r)
        if (cnt[r] < 4 && ((a.job[r] & 15) == 0 || ((a.job[q] >> 11 & 1) ? r < 16 : r < 8 && r + 4 <= q)) && (t < 0 || cnt[r] - base[r] < cnt[t] - base[t])) t = r;
      if (t < 0 || cnt[t] >= 4) { fprintf(stderr, "bad unpack list\n"); return; }
      a.unp[t] |= (unsigned)(0x80 | (q << 2) | u) << (8 * cnt[t]++);
    }
  }
  gemm_a2a<<<256, 256, LDS_BYTES, stream>>>(a);
}
```
